```python
import numpy as np

kernel_name = "gpu_gemm_a2a_m4096_k4096_n16384_f32_silu"


ABSMAX_RTOL = 0.001

COMPUTE_DTYPE = "float32"
EPILOGUE = "silu"
IN_QUANT = None
OUT_QUANT = None
IN_DTYPES = {"x": "float32", "w_mat": "float32"}
WORLD_SIZE = 8

def setup_inputs():

    x = np.random.randn(4096, 4096).astype(np.float32)
    w_mat = (np.random.randn(4096, 16384) / np.sqrt(4096)).astype(np.float32)
    return {"x": x, "w_mat": w_mat}

def _epilogue(y):
    return (y / (1.0 + np.exp(-np.clip(y, -60.0, 60.0)))).astype(np.float32)

def reference(x, w_mat):
    y = (x @ w_mat).astype(np.float32)
    return _epilogue(y)
```

```cpp
#include <hip/hip_runtime.h>
#include <rccl/rccl.h>
#include <cstdio>
#include <cstdint>
#include <type_traits>

#define HIP_CHECK(expr) do { hipError_t _e = (expr); if (_e != hipSuccess) { \
    fprintf(stderr, "HIP error %s at %s:%d\n", hipGetErrorString(_e), __FILE__, __LINE__); \
    return; } } while (0)

namespace {

typedef float f32x4 __attribute__((ext_vector_type(4)));
typedef _Float16 f16x8 __attribute__((ext_vector_type(8)));
typedef unsigned u32x4 __attribute__((ext_vector_type(4)));
#define LAS __attribute__((address_space(3)))

constexpr int NR = 8;
constexpr int MT = 4096, KD = 4096, NT = 16384;
constexpr int MS = MT / NR;
constexpr int NS = NT / NR;
constexpr int BM = 256, BN = 128, BK = 32;
constexpr int NSTEP = KD / BK;
constexpr int NSLOT = 6;
constexpr int A_BYTES = BM * BK * 2;
constexpr int E_BYTES = 2048;
constexpr int B_BYTES = BN * BK * 2;
constexpr int SLOT_BYTES = A_BYTES + E_BYTES + B_BYTES;
constexpr int RING_BYTES = NSLOT * SLOT_BYTES;
constexpr int CTL_OFF = RING_BYTES;
constexpr int LDS_BYTES = RING_BYTES + 128;
constexpr int NCONS = 8;
constexpr int NWAVES = NCONS + 4;
constexpr int NTHREADS = NWAVES * 64;
constexpr int NBLK = 256;
constexpr int NCB = 16;
constexpr int NMT = 15;
#define CWPB 4
#define WATCH_IN_COMM 0
constexpr int NDB = NCB - WATCH_IN_COMM;
constexpr int NCW = NDB * CWPB;
constexpr int NFRAG = (MS / 16) * NSTEP;
#define POLL_SLEEP 2
#define STAGGER_PS 17000
#define GLDS_BITS "sc1"
#define FREE_SLEEP 1
#define LOADER_PRIO 2
#define LDEPTH 4
#define NWW 8
#define WATCH_SLEEP 1
#define POLL_BITS "sc0 sc1"

constexpr size_t EPOCH_OFF = 0;
constexpr size_t TMO_OFF = 16384;
constexpr size_t TICK_OFF = 16384 + 256;
constexpr size_t CNT_OFF = 32768;
constexpr size_t FLAGW_OFF = 65536;
constexpr size_t FLAGX_OFF = 65536 + 16384;
constexpr size_t DBG_OFF = 131072;
constexpr size_t XS_OFF = 1u << 20;
constexpr size_t XS_SLOT = (size_t)MS * KD * 2;
constexpr size_t W16_OFF = XS_OFF + 2 * NR * XS_SLOT;
constexpr size_t WS_NEED = W16_OFF + (size_t)KD * NS * 2;
constexpr unsigned long long DEADLINE_TICKS = 100000000ull;

struct Params {
    const float* x;
    const float* w;
    float* out;
    char* ws[NR];
    int rank;
};

__device__ __forceinline__ unsigned long long rt() { return __builtin_amdgcn_s_memrealtime(); }
__device__ __forceinline__ void lds_st(unsigned addr, unsigned v) {
    asm volatile("ds_write_b32 %0, %1" : : "v"(addr), "v"(v) : "memory");
}
__device__ __forceinline__ u32x4 lds_ld4(unsigned addr) {
    u32x4 v;
    asm volatile("ds_read_b128 %0, %1\n\ts_waitcnt lgkmcnt(0)" : "=&v"(v) : "v"(addr) : "memory");
    return v;
}
__device__ __forceinline__ void glds4k(const void* gsrc, unsigned lds_dst) {
    unsigned keep;
    asm volatile("s_mov_b32 %0, m0\n\ts_mov_b32 m0, %2\n\ts_nop 0\n\t"
                 "global_load_lds_dwordx4 %1, off " GLDS_BITS "\n\t"
                 "global_load_lds_dwordx4 %1, off offset:1024 " GLDS_BITS "\n\t"
                 "global_load_lds_dwordx4 %1, off offset:2048 " GLDS_BITS "\n\t"
                 "global_load_lds_dwordx4 %1, off offset:3072 " GLDS_BITS "\n\t"
                 "s_mov_b32 m0, %0"
                 : "=&s"(keep) : "v"(gsrc), "s"(lds_dst) : "memory");
}
__device__ __forceinline__ void glds1k(const void* gsrc, unsigned lds_dst) {
    unsigned keep;
    asm volatile("s_mov_b32 %0, m0\n\ts_mov_b32 m0, %2\n\ts_nop 0\n\t"
                 "global_load_lds_dwordx4 %1, off " GLDS_BITS "\n\t"
                 "s_mov_b32 m0, %0"
                 : "=&s"(keep) : "v"(gsrc), "s"(lds_dst) : "memory");
}
template <int N> __device__ __forceinline__ void wait_vm() {
    asm volatile("s_waitcnt vmcnt(%0)" : : "i"(N) : "memory");
}
__device__ __forceinline__ void st16_sys(const char* base, unsigned off, f16x8 v) {
    asm volatile("global_store_dwordx4 %0, %1, %2 sc0 sc1\n\ts_nop 1" : : "v"(off), "v"(v), "s"(base) : "memory");
}
__device__ __forceinline__ void st16_dev(const char* base, unsigned off, f16x8 v) {
    asm volatile("global_store_dwordx4 %0, %1, %2 sc1\n\ts_nop 1" : : "v"(off), "v"(v), "s"(base) : "memory");
}
__device__ __forceinline__ void st_out(float* p, f32x4 v) {
    asm volatile("global_store_dwordx4 %0, %1, off sc1\n\ts_nop 1" : : "v"(p), "v"(v) : "memory");
}
__device__ __forceinline__ const char* uni_ptr(const char* p) {
    const unsigned long long u = (unsigned long long)p;
    const unsigned lo = __builtin_amdgcn_readfirstlane((unsigned)u), hi = __builtin_amdgcn_readfirstlane((unsigned)(u >> 32));
    return (const char*)(((unsigned long long)hi << 32) | lo);
}
__device__ __forceinline__ float silu(float y) {
    const float c = __builtin_amdgcn_fmed3f(y, -60.0f, 60.0f);
    const float e = __builtin_amdgcn_exp2f(c * -1.44269504088896340736f);
    return y * __builtin_amdgcn_rcpf(1.0f + e);
}

#define STAMP(idx) do { } while (0)
#define ACC_DECL(n) do { } while (0)
#define ACC_BEG(n) do { } while (0)
#define ACC_END(n) do { } while (0)
#define ACC_OUT(n, idx) do { } while (0)

template <bool ISX, bool EXTRA>
__device__ __forceinline__ void loader(const char* src, const char* esrc, unsigned stride, unsigned lds_base,
                                       unsigned lds_off, unsigned elds_off, unsigned full_addr, const unsigned* flag0, const unsigned* flag1,
                                       unsigned ep, unsigned* tmo, int lane, char* myws, int stamp0) {
    constexpr int PIECES = 8 + (EXTRA ? 1 : 0);
    constexpr int DEPTH = LDEPTH;
    static_assert(PIECES * DEPTH <= 60 && DEPTH <= 5, "vmcnt is 6 bits");
    const unsigned ctl = lds_base + CTL_OFF;
    const char* lsrc = src + lane * 16;
    const char* le = esrc + lane * 16;
    const unsigned* pollp = (lane < 32 ? flag0 : flag1) + (lane & 31);
    int pub = 0, wm = 0, fr = NSLOT;
    int slot = 0;
    ACC_DECL(a_flag); ACC_DECL(a_free); ACC_DECL(a_vm); ACC_DECL(a_iss);
    for (int s = 0; s < NSTEP; ++s) {
        if (s >= wm) {
            ACC_BEG(a_flag);
            const unsigned long long t0 = rt();
            const unsigned* pp = pollp + (s & ~31);
            unsigned spins = 0;
            while (true) {
                unsigned fl;
                if (ISX) asm volatile("global_load_dword %0, %1, off " POLL_BITS : "=&v"(fl) : "v"(pp) : "memory");
                else     asm volatile("global_load_dword %0, %1, off sc1" : "=&v"(fl) : "v"(pp) : "memory");
                if (pub < s) { wait_vm<1>(); lds_st(full_addr, (unsigned)s); pub = s; }
                asm volatile("s_waitcnt vmcnt(0)" : "+v"(fl) : : "memory");
                const unsigned long long m = __ballot((int)(fl - ep) >= 0);
                const unsigned both = (unsigned)m & (unsigned)(m >> 32);
                wm = (s & ~31) + (both == 0xffffffffu ? 32 : __builtin_ctz(~both));
                if (wm > s) break;
                if ((++spins & 63u) == 0u) {
                    const unsigned tv = __builtin_amdgcn_readfirstlane(__hip_atomic_load(tmo, __ATOMIC_RELAXED, __HIP_MEMORY_SCOPE_AGENT));
                    if (tv == ep || rt() - t0 > DEADLINE_TICKS) {
                        if (lane == 0) __hip_atomic_store(tmo, ep, __ATOMIC_RELAXED, __HIP_MEMORY_SCOPE_AGENT);
                        lds_st(ctl + 12, 1u);
                        wait_vm<0>();
                        wm = NSTEP;
                        break;
                    }
                }
                __builtin_amdgcn_s_sleep(POLL_SLEEP);
            }
            if (s > 0) ACC_END(a_flag);
        }
        if (s >= fr) {
            ACC_BEG(a_free);
            while (true) {
                const u32x4 d0 = lds_ld4(ctl + 64), d1 = lds_ld4(ctl + 80);
                unsigned mn = min(min(min(d0.x, d0.y), min(d0.z, d0.w)), min(min(d1.x, d1.y), min(d1.z, d1.w)));
                fr = (int)__builtin_amdgcn_readfirstlane(mn) + NSLOT;
                if (fr > s) break;
                if (pub < s) {
                    const int infl = s - pub;
                    if (infl == 1) wait_vm<0>(); else if (infl == 2) wait_vm<PIECES>(); else if (infl == 3) wait_vm<2 * PIECES>(); else wait_vm<3 * PIECES>();
                    ++pub; lds_st(full_addr, (unsigned)pub);
                } else {
                    __builtin_amdgcn_s_sleep(FREE_SLEEP);
                }
            }
            ACC_END(a_free);
        }
        ACC_BEG(a_iss);
        const unsigned dst = lds_base + (unsigned)slot * SLOT_BYTES;
        const size_t go = (size_t)s * stride;
        glds4k(lsrc + go, dst + lds_off);
        glds4k(lsrc + go + 4096, dst + lds_off + 4096);
        if (EXTRA) glds1k(le + go, dst + elds_off);
        slot = (slot == NSLOT - 1) ? 0 : slot + 1;
        ACC_END(a_iss);
        if (s + 1 - pub >= DEPTH) { ACC_BEG(a_vm); wait_vm<PIECES * (DEPTH - 1)>(); ACC_END(a_vm); ++pub; lds_st(full_addr, (unsigned)pub); }
    }
    wait_vm<0>();
    lds_st(full_addr, (unsigned)NSTEP);
    if (stamp0 <= 10) { ACC_OUT(a_flag, stamp0 == 8 ? 20 : 24); ACC_OUT(a_free, stamp0 == 8 ? 21 : 25); ACC_OUT(a_vm, stamp0 == 8 ? 22 : 26); ACC_OUT(a_iss, stamp0 == 8 ? 23 : 27); }
}

__device__ __forceinline__ void watcher(const Params& p, char* myws, int rank, int lane, unsigned ep, int w, int nw) {
    unsigned* tmo = (unsigned*)(myws + TMO_OFF);
    const unsigned* cntp = (const unsigned*)(myws + CNT_OFF);
    unsigned* flagp = (unsigned*)(p.ws[(rank + 1 + lane) & (NR - 1)] + FLAGX_OFF) + rank * 2 * NSTEP;
    const unsigned need = ep * 16u;
    const unsigned long long t0 = rt();
    unsigned spins = 0;
    for (unsigned o = w; o < 2u * NSTEP; o += nw) {
        while (true) {
            unsigned c;
            asm volatile("global_load_dword %0, %1, off sc1\n\ts_waitcnt vmcnt(0)" : "=&v"(c) : "v"(cntp + o * 32u) : "memory");
            if ((int)(__builtin_amdgcn_readfirstlane(c) - need) >= 0) break;
            if ((++spins & 255u) == 0u && rt() - t0 > DEADLINE_TICKS) {
                if (lane == 0) __hip_atomic_store(tmo, ep, __ATOMIC_RELAXED, __HIP_MEMORY_SCOPE_AGENT);
                return;
            }
            __builtin_amdgcn_s_sleep(WATCH_SLEEP);
        }
        if (lane < NR - 1)
            asm volatile("global_store_dword %0, %1, off sc0 sc1" : : "v"(flagp + (o & 1u) * NSTEP + (o >> 1)), "v"(ep) : "memory");
    }
}

struct Frags { f16x8 a[4], b[4], e; };

__global__ void __launch_bounds__(NTHREADS, 3) fused_kernel(Params p) {
    __shared__ __attribute__((aligned(1024))) char lds[LDS_BYTES];
    const int tid = threadIdx.x;
    const int lane = tid & 63;
    const int wave = __builtin_amdgcn_readfirstlane(tid >> 6);
    const int blk = blockIdx.x;
    const int rank = p.rank;
    char* const myws = p.ws[rank & (NR - 1)];
    const int fr16 = lane & 15, fq = lane >> 4;
    unsigned* const epw = (unsigned*)(myws + EPOCH_OFF) + blk * 16;
    const unsigned lds_base = (unsigned)(size_t)(LAS char*)lds;

    if (blk < NCB) {
        if (wave >= CWPB) return;
        const int cw = wave * NDB + blk;
        const float* xl = p.x + (size_t)fr16 * KD + 4 * fq;
        auto ld = [&](f32x4& a, f32x4& b, unsigned f) {
            f = min(f, (unsigned)NFRAG - 1u);
            const float* xp = xl + (size_t)((f & 31u) * 16u) * KD + (f >> 5) * BK;
            asm volatile("global_load_dwordx4 %0, %2, off\n\tglobal_load_dwordx4 %1, %2, off offset:64" : "=&v"(a), "=&v"(b) : "v"(xp) : "memory");
        };
        f32x4 a0, a1, b0, b1;
        ld(a0, a1, (unsigned)cw);
        ld(b0, b1, (unsigned)(cw + NCW));
        const unsigned ep = __builtin_amdgcn_readfirstlane(__hip_atomic_load(epw + wave, __ATOMIC_RELAXED, __HIP_MEMORY_SCOPE_AGENT)) + 1u;
        asm volatile("s_waitcnt vmcnt(0)" : "+v"(a0), "+v"(a1), "+v"(b0), "+v"(b1) : : "memory");
        if (lane == 0) __hip_atomic_store(epw + wave, ep, __ATOMIC_RELAXED, __HIP_MEMORY_SCOPE_AGENT);
        const size_t slot_off = XS_OFF + ((size_t)(ep & 1u) * NR + rank) * XS_SLOT;
        const char* dst[NR - 1];
#pragma unroll
        for (int q = 0; q < NR - 1; ++q) {
            const unsigned pr = (unsigned)(rank + 1 + (q + cw) % (NR - 1)) & (NR - 1);
            dst[q] = uni_ptr(p.ws[pr] + slot_off);
        }
        const char* ldst = uni_ptr(myws + slot_off);
        unsigned* tick = (unsigned*)(myws + TICK_OFF);
        unsigned* cntp = (unsigned*)(myws + CNT_OFF);
        const unsigned tbase = (ep - 1u) * (unsigned)(NFRAG - 2 * NCW) - 3u * (unsigned)NCW;
        unsigned f0 = (unsigned)cw, f1 = f0 + NCW, f2 = f1 + NCW, fp = 0xffffffffu;
        if (wave == 0) STAMP(16);
        auto round = [&](f32x4& c0, f32x4& c1, f32x4& n0, f32x4& n1) {
            f16x8 h;
#pragma unroll
            for (int e = 0; e < 4; ++e) { h[e] = (_Float16)c0[e]; h[4 + e] = (_Float16)c1[e]; }
            asm volatile("" : "+v"(h) : : "memory");
            unsigned t = 0;
            const bool draw = f2 < (unsigned)NFRAG;
            {
                unsigned long long save;
                const unsigned msk = __builtin_amdgcn_readfirstlane(draw ? 1u : 0u);
                asm volatile("s_mov_b64 %1, exec\n\ts_mov_b32 exec_lo, %4\n\ts_mov_b32 exec_hi, 0\n\tglobal_atomic_add %0, %2, %3, off sc0\n\ts_mov_b64 exec, %1"
                             : "+v"(t), "=&s"(save) : "v"(tick), "v"(1u), "s"(msk) : "memory");
            }
            const unsigned off = ((((f0 >> 4) & 1u) * NSTEP + (f0 >> 5)) * 16u + (f0 & 15u)) * 1024u + (unsigned)lane * 16u;
#pragma unroll
            for (int q = 0; q < NR - 1; ++q) {
                st16_sys(dst[q], off, h);
            }
            ld(c0, c1, f2);
            asm volatile("s_waitcnt vmcnt(9)" : "+v"(t), "+v"(n0), "+v"(n1) : : "memory");
            if (fp != 0xffffffffu && lane == 0) asm volatile("global_atomic_add %0, %1, off" : : "v"(cntp + (fp >> 4) * 32u), "v"(1u) : "memory");
            fp = f0; f0 = f1; f1 = f2;
            f2 = draw ? __builtin_amdgcn_readfirstlane(t) - tbase : 0xffffffffu;
        };
        while (true) {
            round(a0, a1, b0, b1);
            if (f0 >= (unsigned)NFRAG) break;
            round(b0, b1, a0, a1);
            if (f0 >= (unsigned)NFRAG) break;
        }
        asm volatile("s_waitcnt vmcnt(0)" : "+v"(a0), "+v"(a1), "+v"(b0), "+v"(b1) : : "memory");
        if (lane == 0) asm volatile("global_atomic_add %0, %1, off" : : "v"(cntp + (fp >> 4) * 32u), "v"(1u) : "memory");
        STAMP(17 + wave);
        return;
    }

    const unsigned ctl = lds_base + CTL_OFF;
    const int xcd = blk & 7;
    const int t = (xcd >> 1) * 30 + ((blk >> 3) - 2);
    const int im = t >> 3;
    const int jn = (xcd & 1) * 8 + (t & 7);
    const int src = (rank + ((im + 1) >> 1)) & (NR - 1);
    const int half = (im == 0) ? 0 : ((im + 1) & 1);
    const int ef0 = (im == 0) ? 14 : im - 1;
    const float* wl = p.w + (size_t)(4 * fq) * NT + (size_t)rank * NS + jn * BN + 4 * (4 * (fr16 & 3) + (fr16 >> 2));
    const char* wdst = uni_ptr(myws + W16_OFF + (size_t)jn * NSTEP * B_BYTES);
    unsigned* fwp = (unsigned*)(myws + FLAGW_OFF) + jn * 2 * NSTEP;

    if (wave < NCONS) {
        STAMP(0);
        const int u = im * NCONS + wave;
        f32x4 wv[8];
        {
            const float* wu = wl + (size_t)(BK * (u >> 1)) * NT + 64 * (u & 1);
#pragma unroll
            for (int e = 0; e < 8; ++e) wv[e] = *(const f32x4*)(wu + (size_t)((e >> 2) * 16 + (e & 3)) * NT);
        }
        asm volatile("" ::: "memory");
        __builtin_amdgcn_s_barrier();
        asm volatile("" ::: "memory");
        const unsigned ep = __builtin_amdgcn_readfirstlane(lds_ld4(ctl + 16).x);
        {
            f16x8 h[4];
#pragma unroll
            for (int i = 0; i < 4; ++i)
#pragma unroll
                for (int e = 0; e < 8; ++e) h[i][e] = (_Float16)wv[e][i];
            asm volatile("" : "+v"(h[0]), "+v"(h[1]), "+v"(h[2]), "+v"(h[3]) : : "memory");
#pragma unroll
            for (int i = 0; i < 4; ++i) st16_dev(wdst, (unsigned)((u >> 1) * B_BYTES + (u & 1) * 4096 + i * 1024 + lane * 16), h[i]);
            wait_vm<0>();
            if (lane == 0) __hip_atomic_store(fwp + (u & 1) * NSTEP + (u >> 1), ep, __ATOMIC_RELAXED, __HIP_MEMORY_SCOPE_AGENT);
        }
        STAMP(1);
        const int wm_ = wave & 3, wn_ = wave >> 2;
        const int ek = wm_ >> 1;
        const bool has_e = ((wm_ & 1) == wn_) && (ek == 0 || im == 0);
        f32x4 acc[4][4], ex[4];
#pragma unroll
        for (int j = 0; j < 4; ++j) {
#pragma unroll
            for (int i = 0; i < 4; ++i) acc[i][j] = (f32x4){0.f, 0.f, 0.f, 0.f};
            ex[j] = (f32x4){0.f, 0.f, 0.f, 0.f};
        }
        const LAS char* abase = (const LAS char*)lds + wm_ * 4096 + lane * 16;
        const LAS char* ebase = (const LAS char*)lds + A_BYTES + ek * 1024 + lane * 16;
        const LAS char* bbase = (const LAS char*)lds + A_BYTES + E_BYTES + wn_ * 4096 + lane * 16;
        const LAS u32x4* ctlp = (const LAS u32x4*)((const LAS char*)lds + CTL_OFF);
        const unsigned done_addr = ctl + 64 + wave * 4;
        ACC_DECL(a_wa); ACC_DECL(a_tot);
        auto spin_avail = [&](int s) {
            ACC_BEG(a_wa);
            while (true) {
                const u32x4 f = lds_ld4(ctl);
                if ((int)__builtin_amdgcn_readfirstlane(min(min(f.x, f.y), f.z)) > s) break;
                __builtin_amdgcn_s_sleep(1);
            }
            if (s > 0) ACC_END(a_wa);
        };
        auto run = [&](auto he) {
            constexpr bool HE = decltype(he)::value;
            Frags F0, F1;
            u32x4 cw_;
            auto load = [&](Frags& F, int slot) {
                const int so = slot * SLOT_BYTES;
#pragma unroll
                for (int j = 0; j < 4; ++j) F.b[j] = *(const LAS f16x8*)(bbase + so + j * 1024);
#pragma unroll
                for (int i = 0; i < 4; ++i) F.a[i] = *(const LAS f16x8*)(abase + so + i * 1024);
                if (HE) F.e = *(const LAS f16x8*)(ebase + so);
                cw_ = *(const volatile LAS u32x4*)ctlp;
            };
            auto mma = [&](const Frags& F) {
#pragma unroll
                for (int i = 0; i < 4; ++i)
#pragma unroll
                    for (int j = 0; j < 4; ++j)
                        acc[i][j] = __builtin_amdgcn_mfma_f32_16x16x32_f16(F.b[j], F.a[i], acc[i][j], 0, 0, 0);
                if (HE) {
#pragma unroll
                    for (int j = 0; j < 4; ++j) ex[j] = __builtin_amdgcn_mfma_f32_16x16x32_f16(F.b[j], F.e, ex[j], 0, 0, 0);
                }
            };
            auto step = [&](Frags& Fc, Frags& Fn, int s, int nslot) {
                __builtin_amdgcn_s_waitcnt(0xC07F);
                lds_st(done_addr, (unsigned)(s + 1));
                const int av = (int)__builtin_amdgcn_readfirstlane(min(min(cw_.x, cw_.y), cw_.z));
                if (av <= s + 1 && s + 1 < NSTEP) spin_avail(s + 1);
                __builtin_amdgcn_sched_barrier(0);
                load(Fn, nslot);
                mma(Fc);
#pragma unroll
                for (int g = 0; g < (HE ? 10 : 9); ++g) {
                    __builtin_amdgcn_sched_group_barrier(0x008, 1, 0);
                    __builtin_amdgcn_sched_group_barrier(0x100, 1, 0);
                }
                __builtin_amdgcn_sched_barrier(0);
            };
            spin_avail(0);
            load(F0, 0);
            if (wave == 0) STAMP(2);
            ACC_BEG(a_tot);
            int slot = 0;
            for (int s = 0; s < NSTEP; s += 2) {
                step(F0, F1, s, slot + 1);
                slot = (slot == NSLOT - 2) ? 0 : slot + 2;
                step(F1, F0, s + 1, slot);
            }
            ACC_END(a_tot);
        };
        if (has_e) run(std::integral_constant<bool, true>{}); else run(std::integral_constant<bool, false>{});
        if (wave == 0) STAMP(3);
        if (wave == 0) { ACC_OUT(a_wa, 28); ACC_OUT(a_tot, 30); }
        const bool poison = __builtin_amdgcn_readfirstlane(lds_ld4(ctl).w) != 0u;
        const float nanv = __builtin_nanf("");
        float* oc = p.out + jn * BN + wn_ * 64 + 4 * fq;
        float* op = oc + (size_t)(src * MS + half * BM + wm_ * 64 + fr16) * NS;
#pragma unroll
        for (int i = 0; i < 4; ++i)
#pragma unroll
            for (int r = 0; r < 4; ++r) {
                f32x4 v;
#pragma unroll
                for (int j = 0; j < 4; ++j) v[j] = silu(acc[i][j][r]);
                if (poison) v = (f32x4){nanv, nanv, nanv, nanv};
                st_out(op + (size_t)(16 * i) * NS + 16 * r, v);
            }
        if (has_e) {
            float* oe = oc + (size_t)(rank * MS + BM + (ef0 + ek) * 16 + fr16) * NS;
#pragma unroll
            for (int r = 0; r < 4; ++r) {
                f32x4 v;
#pragma unroll
                for (int j = 0; j < 4; ++j) v[j] = silu(ex[j][r]);
                if (poison) v = (f32x4){nanv, nanv, nanv, nanv};
                st_out(oe + 16 * r, v);
            }
        }
    } else {
        __builtin_amdgcn_s_setprio(LOADER_PRIO);
        unsigned ep;
        if (wave == NCONS) {
            ep = __builtin_amdgcn_readfirstlane(__hip_atomic_load(epw, __ATOMIC_RELAXED, __HIP_MEMORY_SCOPE_AGENT)) + 1u;
            if (lane < 32) lds_st(ctl + lane * 4, lane == 4 ? ep : 0u);
            if (lane == 0) __hip_atomic_store(epw, ep, __ATOMIC_RELAXED, __HIP_MEMORY_SCOPE_AGENT);
            asm volatile("s_waitcnt lgkmcnt(0)" ::: "memory");
        }
        asm volatile("" ::: "memory");
        __builtin_amdgcn_s_barrier();
        asm volatile("" ::: "memory");
        ep = __builtin_amdgcn_readfirstlane(lds_ld4(ctl + 16).x);
        unsigned* tmo = (unsigned*)(myws + TMO_OFF);
        if (wave == NCONS + 3) {
            if (im == 0) { watcher(p, myws, rank, lane, ep, jn, 16); STAMP(15); return; }
            if (im == 0) return;
            {
                const char* ldst = uni_ptr(myws + XS_OFF + ((size_t)(ep & 1u) * NR + rank) * XS_SLOT);
                unsigned* lflag = (unsigned*)(myws + FLAGX_OFF) + rank * 2 * NSTEP;
                for (int o = (im - 1) * 16 + jn; o < 2 * NSTEP; o += 14 * 16) {
                    const int unit = (o & 1) * NSTEP + (o >> 1);
                    const float* xu = p.x + (size_t)((o & 1) * BM + fr16) * KD + (o >> 1) * BK + 4 * fq;
#pragma unroll
                    for (int hh = 0; hh < 2; ++hh) {
                        f32x4 xv[8][2];
#pragma unroll
                        for (int m = 0; m < 8; ++m) {
                            xv[m][0] = *(const f32x4*)(xu + (size_t)((hh * 8 + m) * 16) * KD);
                            xv[m][1] = *(const f32x4*)(xu + (size_t)((hh * 8 + m) * 16) * KD + 16);
                        }
                        f16x8 h[8];
#pragma unroll
                        for (int m = 0; m < 8; ++m)
#pragma unroll
                            for (int e = 0; e < 4; ++e) { h[m][e] = (_Float16)xv[m][0][e]; h[m][4 + e] = (_Float16)xv[m][1][e]; }
                        asm volatile("" : "+v"(h[0]), "+v"(h[1]), "+v"(h[2]), "+v"(h[3]), "+v"(h[4]), "+v"(h[5]), "+v"(h[6]), "+v"(h[7]) : : "memory");
#pragma unroll
                        for (int m = 0; m < 8; ++m) st16_dev(ldst, (unsigned)((unit * 16 + hh * 8 + m) * 1024 + lane * 16), h[m]);
                    }
                    wait_vm<0>();
                    if (lane == 0) __hip_atomic_store(lflag + unit, ep, __ATOMIC_RELAXED, __HIP_MEMORY_SCOPE_AGENT);
                }
            }
            STAMP(5);
            for (int st = 60 + im - 1; st < NSTEP; st += NMT - 1) {
                f32x4 wv[2][8];
#pragma unroll
                for (int pp = 0; pp < 2; ++pp)
#pragma unroll
                    for (int e = 0; e < 8; ++e)
                        wv[pp][e] = *(const f32x4*)(wl + (size_t)(BK * st + (e >> 2) * 16 + (e & 3)) * NT + 64 * pp);
                f16x8 h[8];
#pragma unroll
                for (int pp = 0; pp < 2; ++pp)
#pragma unroll
                    for (int i = 0; i < 4; ++i)
#pragma unroll
                        for (int e = 0; e < 8; ++e) h[4 * pp + i][e] = (_Float16)wv[pp][e][i];
                asm volatile("" : "+v"(h[0]), "+v"(h[1]), "+v"(h[2]), "+v"(h[3]), "+v"(h[4]), "+v"(h[5]), "+v"(h[6]), "+v"(h[7]) : : "memory");
#pragma unroll
                for (int i = 0; i < 8; ++i) st16_dev(wdst, (unsigned)(st * B_BYTES + i * 1024 + lane * 16), h[i]);
                wait_vm<0>();
                if (lane < 2) __hip_atomic_store(fwp + lane * NSTEP + st, ep, __ATOMIC_RELAXED, __HIP_MEMORY_SCOPE_AGENT);
            }
            STAMP(14);
            return;
        }
        if (wave == NCONS) {
            const char* ws16 = myws + W16_OFF + (size_t)jn * NSTEP * B_BYTES;
            const unsigned* fw = (const unsigned*)(myws + FLAGW_OFF) + jn * 2 * NSTEP;
            loader<false, false>(ws16, ws16, B_BYTES, lds_base, A_BYTES + E_BYTES, 0, ctl + 8, fw, fw + NSTEP, ep, tmo, lane, myws, 8);
        } else {
            const int la = wave - NCONS - 1;
            const char* xsb = myws + XS_OFF + (size_t)(ep & 1u) * NR * XS_SLOT;
            const char* xs = xsb + (size_t)src * XS_SLOT + (size_t)half * NSTEP * A_BYTES + la * 8192;
            const char* xe = xsb + (size_t)rank * XS_SLOT + (size_t)NSTEP * A_BYTES + (ef0 + la) * 1024;
            const unsigned* fx = (const unsigned*)(myws + FLAGX_OFF);
            const unsigned* f0 = fx + (src * 2 + half) * NSTEP;
            if (la == 0 || im == 0)
                loader<true, true>(xs, xe, A_BYTES, lds_base, la * 8192, A_BYTES + la * 1024, ctl + 4 * la, f0, fx + (rank * 2 + 1) * NSTEP, ep, tmo, lane, myws, 10 + 2 * la);
            else
                loader<true, false>(xs, xs, A_BYTES, lds_base, la * 8192, 0, ctl + 4 * la, f0, f0, ep, tmo, lane, myws, 10 + 2 * la);
        }
    }
}

}

extern "C" void kernel_launch(int rank, int nranks, ncclComm_t comm,
                              hipStream_t stream,
                              void* const* d_in, const int* in_sizes, int n_in,
                              void* d_out, int out_size,
                              const int* in_dtypes, int out_dtype,
                              void* const* d_out_all,
                              void* const* d_ws_all, size_t ws_bytes) {
    (void)comm; (void)in_dtypes; (void)out_dtype; (void)d_out_all;
    if (nranks != NR || (unsigned)rank >= (unsigned)NR || n_in < 2 || ws_bytes < WS_NEED ||
        in_sizes[0] != MS * KD || in_sizes[1] != KD * NT || out_size != MT * NS) {
        fprintf(stderr, "kernel_launch: unexpected problem shape\n");
        return;
    }
    Params p;
    p.x = (const float*)d_in[0];
    p.w = (const float*)d_in[1];
    p.out = (float*)d_out;
    for (int i = 0; i < NR; ++i) p.ws[i] = (char*)d_ws_all[i];
    p.rank = rank;
    hipLaunchKernelGGL(fused_kernel, dim3(NBLK), dim3(NTHREADS), 0, stream, p);
    HIP_CHECK(hipGetLastError());
}
```

```cpp
#include <hip/hip_runtime.h>
#include <rccl/rccl.h>
#include <cstdio>
#include <cstdint>
#include <type_traits>

#define HIP_CHECK(expr) do { hipError_t _e = (expr); if (_e != hipSuccess) { \
    fprintf(stderr, "HIP error %s at %s:%d\n", hipGetErrorString(_e), __FILE__, __LINE__); \
    return; } } while (0)

namespace {

typedef float f32x4 __attribute__((ext_vector_type(4)));
typedef _Float16 f16x8 __attribute__((ext_vector_type(8)));
typedef unsigned u32x4 __attribute__((ext_vector_type(4)));
#define LAS __attribute__((address_space(3)))

constexpr int NR = 8;
constexpr int MT = 4096, KD = 4096, NT = 16384;
constexpr int NTW = NT, KDX = KD;
constexpr int MS = MT / NR;
constexpr int NS = NT / NR;
constexpr int BM = 256, BN = 128, BK = 32;
constexpr int NSTEP = KD / BK;
constexpr int NSLOT = 6;
constexpr int A_BYTES = BM * BK * 2;
constexpr int E_BYTES = 2048;
constexpr int B_BYTES = BN * BK * 2;
constexpr int SLOT_BYTES = A_BYTES + E_BYTES + B_BYTES;
constexpr int RING_BYTES = NSLOT * SLOT_BYTES;
constexpr int CTL_OFF = RING_BYTES;
constexpr int LDS_BYTES = RING_BYTES + 128;
constexpr int NCONS = 8;
constexpr int NWAVES = NCONS + 4;
constexpr int NTHREADS = NWAVES * 64;
constexpr int NBLK = 256;
constexpr int NCB = 16;
constexpr int NMT = 15;
#define CWPB 4
#define WATCH_IN_COMM 0
constexpr int NDB = NCB - WATCH_IN_COMM;
constexpr int NCW = NDB * CWPB;
constexpr int NFRAG = (MS / 16) * NSTEP;
#define POLL_SLEEP 2
#define STAGGER_PS 17000
#define GLDS_BITS "sc1"
#define FREE_SLEEP 1
#define LOADER_PRIO 2
#define LDEPTH 3
#define NWW 8
#define WATCH_SLEEP 1
#define CV_BY_XCD 0
#define NT_W 1
#define NT_X 0
#define CX_BITS ""
#define CVST_BITS "sc1"
#define RST_BITS "sc0 sc1"
#define OUT_BITS "sc1"
#define CV_WAVES 4
#define HELPER_GAP 0
#define EP_WARM 0
#define OUT_TOUCH 0
#define DIRECT_T 0
#define START_DELAY 0
#define HELP_DELAY 0
#define CWPB_KEEP 3
#define RETIRE_F 3584
#define POLL_LONG 32
#define POLL_TAIL 12
#define ONE_POLLER 1
#define POLL_BITS "sc0 sc1"

constexpr size_t EPOCH_OFF = 0;
constexpr size_t TMO_OFF = 16384;
constexpr size_t TICK_OFF = 16384 + 256;
constexpr size_t CNT_OFF = 32768;
constexpr size_t FLAGW_OFF = 65536;
constexpr size_t FLAGX_OFF = 65536 + 16384;
constexpr size_t DBG_OFF = 131072;
constexpr size_t XS_OFF = 1u << 20;
constexpr size_t XS_SLOT = (size_t)MS * KD * 2;
constexpr size_t W16_OFF = XS_OFF + 2 * NR * XS_SLOT;
constexpr size_t WS_NEED = W16_OFF + (size_t)KD * NS * 2;
constexpr unsigned long long DEADLINE_TICKS = 100000000ull;

struct Params {
    const float* x;
    const float* w;
    float* out;
    char* ws[NR];
    int rank;
};

__device__ __forceinline__ f32x4 ld_w(const float* p) {
    return __builtin_nontemporal_load((const f32x4*)p);
}
__device__ __forceinline__ f32x4 ld_x(const float* p) {
    return *(const f32x4*)p;
}
__device__ __forceinline__ unsigned long long rt() { return __builtin_amdgcn_s_memrealtime(); }
__device__ __forceinline__ void lds_st(unsigned addr, unsigned v) {
    asm volatile("ds_write_b32 %0, %1" : : "v"(addr), "v"(v) : "memory");
}
__device__ __forceinline__ u32x4 lds_ld4(unsigned addr) {
    u32x4 v;
    asm volatile("ds_read_b128 %0, %1\n\ts_waitcnt lgkmcnt(0)" : "=&v"(v) : "v"(addr) : "memory");
    return v;
}
__device__ __forceinline__ void glds4k(const void* gsrc, unsigned lds_dst) {
    unsigned keep;
    asm volatile("s_mov_b32 %0, m0\n\ts_mov_b32 m0, %2\n\ts_nop 0\n\t"
                 "global_load_lds_dwordx4 %1, off " GLDS_BITS "\n\t"
                 "global_load_lds_dwordx4 %1, off offset:1024 " GLDS_BITS "\n\t"
                 "global_load_lds_dwordx4 %1, off offset:2048 " GLDS_BITS "\n\t"
                 "global_load_lds_dwordx4 %1, off offset:3072 " GLDS_BITS "\n\t"
                 "s_mov_b32 m0, %0"
                 : "=&s"(keep) : "v"(gsrc), "s"(lds_dst) : "memory");
}
__device__ __forceinline__ void glds1k(const void* gsrc, unsigned lds_dst) {
    unsigned keep;
    asm volatile("s_mov_b32 %0, m0\n\ts_mov_b32 m0, %2\n\ts_nop 0\n\t"
                 "global_load_lds_dwordx4 %1, off " GLDS_BITS "\n\t"
                 "s_mov_b32 m0, %0"
                 : "=&s"(keep) : "v"(gsrc), "s"(lds_dst) : "memory");
}
template <int N> __device__ __forceinline__ void wait_vm() {
    asm volatile("s_waitcnt vmcnt(%0)" : : "i"(N) : "memory");
}
__device__ __forceinline__ void st16_sys(const char* base, unsigned off, f16x8 v) {
    asm volatile("global_store_dwordx4 %0, %1, %2 " RST_BITS "\n\ts_nop 1" : : "v"(off), "v"(v), "s"(base) : "memory");
}
__device__ __forceinline__ void st16_dev(const char* base, unsigned off, f16x8 v) {
    asm volatile("global_store_dwordx4 %0, %1, %2 " CVST_BITS "\n\ts_nop 1" : : "v"(off), "v"(v), "s"(base) : "memory");
}
__device__ __forceinline__ void st_out(float* p, f32x4 v, unsigned live) {
    asm volatile("s_mov_b32 exec_lo, %2\n\ts_mov_b32 exec_hi, %2\n\tglobal_store_dwordx4 %0, %1, off " OUT_BITS "\n\ts_mov_b64 exec, -1\n\ts_nop 0"
                 : : "v"(p), "v"(v), "s"(live) : "memory");
}
__device__ __forceinline__ const char* uni_ptr(const char* p) {
    const unsigned long long u = (unsigned long long)p;
    const unsigned lo = __builtin_amdgcn_readfirstlane((unsigned)u), hi = __builtin_amdgcn_readfirstlane((unsigned)(u >> 32));
    return (const char*)(((unsigned long long)hi << 32) | lo);
}
__device__ __forceinline__ f32x4 sigm4(f32x4 y) {
    f32x4 c, e, r;
#pragma unroll
    for (int i = 0; i < 4; ++i) c[i] = __builtin_amdgcn_fmed3f(y[i], -60.0f, 60.0f);
    c = c * -1.44269504088896340736f;
#pragma unroll
    for (int i = 0; i < 4; ++i) e[i] = __builtin_amdgcn_exp2f(c[i]);
    e = e + 1.0f;
#pragma unroll
    for (int i = 0; i < 4; ++i) r[i] = __builtin_amdgcn_rcpf(e[i]);
    return r;
}

#define STAMP(idx) do { } while (0)
#define ACC_DECL(n) do { } while (0)
#define ACC_BEG(n) do { } while (0)
#define ACC_END(n) do { } while (0)
#define ACC_OUT(n, idx) do { } while (0)

template <bool ISX, bool EXTRA>
__device__ __forceinline__ void loader(const char* src, const char* esrc, unsigned stride, unsigned lds_base,
                                       unsigned lds_off, unsigned elds_off, unsigned full_addr, const unsigned* flag0, const unsigned* flag1,
                                       unsigned ep, unsigned* tmo, int lane, char* myws, int stamp0, bool remote = false, int role = 0) {
    constexpr int PIECES = 8 + (EXTRA ? 1 : 0);
    constexpr int DEPTH = LDEPTH;
    static_assert(PIECES * DEPTH <= 60 && DEPTH <= 5, "vmcnt is 6 bits");
    const unsigned ctl = lds_base + CTL_OFF;
    const char* lsrc = src + lane * 16;
    const char* le = esrc + lane * 16;
    const unsigned* pollp = (lane < 32 ? flag0 : flag1) + (lane & 31);
    int pub = 0, wm = 0, fr = NSLOT;
    int slot = 0;
    ACC_DECL(a_flag); ACC_DECL(a_free); ACC_DECL(a_vm); ACC_DECL(a_iss);
    for (int s = 0; s < NSTEP; ++s) {
        if (s >= wm && role == 2) {
            while (true) {
                unsigned w;
                asm volatile("ds_read_b32 %0, %1 offset:24" : "=&v"(w) : "v"(ctl) : "memory");
                const bool had = pub < s;
                if (had) { wait_vm<0>(); lds_st(full_addr, (unsigned)s); pub = s; }
                asm volatile("s_waitcnt lgkmcnt(0)" : "+v"(w) : : "memory");
                wm = (int)__builtin_amdgcn_readfirstlane(w);
                if (wm > s) break;
                if (!had) __builtin_amdgcn_s_sleep(1);
            }
        }
        if (s >= wm) {
            ACC_BEG(a_flag);
            const unsigned long long t0 = rt();
            const unsigned* pp = pollp + (s & ~31);
            const unsigned thr = (DIRECT_T && remote && (lane < 32 || !EXTRA) && (s & ~31) + (lane & 31) >= NSTEP - DIRECT_T / 2) ? ep * 16u : ep;
            unsigned spins = 0;
            while (true) {
                unsigned fl;
                if (ISX) asm volatile("global_load_dword %0, %1, off " POLL_BITS : "=&v"(fl) : "v"(pp) : "memory");
                else     asm volatile("global_load_dword %0, %1, off sc1" : "=&v"(fl) : "v"(pp) : "memory");
                if (pub < s) { wait_vm<1>(); lds_st(full_addr, (unsigned)s); pub = s; }
                asm volatile("s_waitcnt vmcnt(0)" : "+v"(fl) : : "memory");
                const unsigned long long m = __ballot((int)(fl - thr) >= 0);
                const unsigned both = (unsigned)m & (unsigned)(m >> 32);
                wm = (s & ~31) + (both == 0xffffffffu ? 32 : __builtin_ctz(~both));
                if (wm > s) break;
                if ((++spins & 63u) == 0u) {
                    const unsigned tv = __builtin_amdgcn_readfirstlane(__hip_atomic_load(tmo, __ATOMIC_RELAXED, __HIP_MEMORY_SCOPE_AGENT));
                    if (tv == ep || rt() - t0 > DEADLINE_TICKS) {
                        if (lane == 0) __hip_atomic_store(tmo, ep, __ATOMIC_RELAXED, __HIP_MEMORY_SCOPE_AGENT);
                        lds_st(ctl + 12, 1u);
                        wait_vm<0>();
                        wm = NSTEP;
                        break;
                    }
                }
                if (s < NSTEP - POLL_TAIL) __builtin_amdgcn_s_sleep(POLL_LONG); else __builtin_amdgcn_s_sleep(POLL_SLEEP);
            }
            if (role == 1) lds_st(ctl + 24, (unsigned)wm);
            if (s > 0) ACC_END(a_flag);
        }
        if (s >= fr) {
            ACC_BEG(a_free);
            while (true) {
                const u32x4 d0 = lds_ld4(ctl + 64), d1 = lds_ld4(ctl + 80);
                unsigned mn = min(min(min(d0.x, d0.y), min(d0.z, d0.w)), min(min(d1.x, d1.y), min(d1.z, d1.w)));
                fr = (int)__builtin_amdgcn_readfirstlane(mn) + NSLOT;
                if (fr > s) break;
                if (pub < s) {
                    const int infl = s - pub;
                    if (infl == 1) wait_vm<0>(); else if (infl == 2) wait_vm<PIECES>(); else if (infl == 3) wait_vm<2 * PIECES>(); else wait_vm<3 * PIECES>();
                    ++pub; lds_st(full_addr, (unsigned)pub);
                } else {
                    __builtin_amdgcn_s_sleep(FREE_SLEEP);
                }
            }
            ACC_END(a_free);
        }
        ACC_BEG(a_iss);
        const unsigned dst = lds_base + (unsigned)slot * SLOT_BYTES;
        const size_t go = (size_t)s * stride;
        glds4k(lsrc + go, dst + lds_off);
        glds4k(lsrc + go + 4096, dst + lds_off + 4096);
        if (EXTRA) glds1k(le + go, dst + elds_off);
        slot = (slot == NSLOT - 1) ? 0 : slot + 1;
        ACC_END(a_iss);
        if (s + 1 - pub >= DEPTH) { ACC_BEG(a_vm); wait_vm<PIECES * (DEPTH - 1)>(); ACC_END(a_vm); ++pub; lds_st(full_addr, (unsigned)pub); }
    }
    wait_vm<0>();
    lds_st(full_addr, (unsigned)NSTEP);
    if (stamp0 <= 10) { ACC_OUT(a_flag, stamp0 == 8 ? 20 : 24); ACC_OUT(a_free, stamp0 == 8 ? 21 : 25); ACC_OUT(a_vm, stamp0 == 8 ? 22 : 26); ACC_OUT(a_iss, stamp0 == 8 ? 23 : 27); }
}

__device__ __forceinline__ void watcher(const Params& p, char* myws, int rank, int lane, unsigned ep, int w, int nw) {
    unsigned* tmo = (unsigned*)(myws + TMO_OFF);
    const unsigned* cntp = (const unsigned*)(myws + CNT_OFF);
    unsigned* flagp = (unsigned*)(p.ws[(rank + 1 + lane) & (NR - 1)] + FLAGX_OFF) + rank * 2 * NSTEP;
    const unsigned need = ep * 16u;
    const unsigned long long t0 = rt();
    unsigned spins = 0;
    for (unsigned o = w; o < 2u * NSTEP - DIRECT_T; o += nw) {
        while (true) {
            unsigned c;
            asm volatile("global_load_dword %0, %1, off sc1\n\ts_waitcnt vmcnt(0)" : "=&v"(c) : "v"(cntp + o * 32u) : "memory");
            if ((int)(__builtin_amdgcn_readfirstlane(c) - need) >= 0) break;
            if ((++spins & 255u) == 0u && rt() - t0 > DEADLINE_TICKS) {
                if (lane == 0) __hip_atomic_store(tmo, ep, __ATOMIC_RELAXED, __HIP_MEMORY_SCOPE_AGENT);
                return;
            }
            __builtin_amdgcn_s_sleep(WATCH_SLEEP);
        }
        if (lane < NR - 1)
            asm volatile("global_store_dword %0, %1, off sc0 sc1" : : "v"(flagp + (o & 1u) * NSTEP + (o >> 1)), "v"(ep) : "memory");
    }
}

struct Frags { f16x8 a[4], b[4], e; };

__global__ void __launch_bounds__(NTHREADS, 3) fused_kernel(Params p) {
    __shared__ __attribute__((aligned(1024))) char lds[LDS_BYTES];
    const int tid = threadIdx.x;
    const int lane = tid & 63;
    const int wave = __builtin_amdgcn_readfirstlane(tid >> 6);
    const int blk = blockIdx.x;
    const int rank = p.rank;
    char* const myws = p.ws[rank & (NR - 1)];
    const int fr16 = lane & 15, fq = lane >> 4;
    unsigned* const epw = (unsigned*)(myws + EPOCH_OFF) + blk * 16;
    const unsigned lds_base = (unsigned)(size_t)(LAS char*)lds;

    if (blk < NCB) {
        if (wave >= CWPB) return;
        const int cw = wave * NDB + blk;
        const float* xl = p.x + (size_t)fr16 * KD + 4 * fq;
        auto ld = [&](f32x4& a, f32x4& b, unsigned f) {
            f = min(f, (unsigned)NFRAG - 1u);
            const float* xp = xl + (size_t)((f & 31u) * 16u) * KD + (f >> 5) * BK;
            asm volatile("global_load_dwordx4 %0, %2, off " CX_BITS "\n\tglobal_load_dwordx4 %1, %2, off offset:64 " CX_BITS : "=&v"(a), "=&v"(b) : "v"(xp) : "memory");
        };
        f32x4 a0, a1, b0, b1;
        ld(a0, a1, (unsigned)cw);
        ld(b0, b1, (unsigned)(cw + NCW));
        const unsigned ep = __builtin_amdgcn_readfirstlane(__hip_atomic_load(epw + wave, __ATOMIC_RELAXED, __HIP_MEMORY_SCOPE_AGENT)) + 1u;
        asm volatile("s_waitcnt vmcnt(0)" : "+v"(a0), "+v"(a1), "+v"(b0), "+v"(b1) : : "memory");
        if (lane == 0) __hip_atomic_store(epw + wave, ep, __ATOMIC_RELAXED, __HIP_MEMORY_SCOPE_AGENT);
        const size_t slot_off = XS_OFF + ((size_t)(ep & 1u) * NR + rank) * XS_SLOT;
        const char* dst[NR - 1];
#pragma unroll
        for (int q = 0; q < NR - 1; ++q) {
            const unsigned pr = (unsigned)(rank + 1 + (q + cw) % (NR - 1)) & (NR - 1);
            dst[q] = uni_ptr(p.ws[pr] + slot_off);
        }
        const char* ldst = uni_ptr(myws + slot_off);
        unsigned* tick = (unsigned*)(myws + TICK_OFF);
        unsigned* cntp = (unsigned*)(myws + CNT_OFF);
        const unsigned tbase = (ep - 1u) * (unsigned)(NFRAG - 3 * NCW + NDB * CWPB_KEEP) - 3u * (unsigned)NCW;
        const unsigned flim = wave < CWPB_KEEP ? (unsigned)NFRAG : (unsigned)RETIRE_F;
        unsigned f0 = (unsigned)cw, f1 = f0 + NCW, f2 = f1 + NCW, fp = 0xffffffffu;
        if (wave == 0) STAMP(16);
        unsigned* rflag = (unsigned*)(p.ws[(rank + 1 + lane) & (NR - 1)] + FLAGX_OFF) + rank * 2 * NSTEP;
        auto count = [&](unsigned f) {
            if (DIRECT_T && (f >> 4) >= 2u * NSTEP - DIRECT_T) {
                if (lane < NR - 1) asm volatile("global_atomic_add %0, %1, off sc1" : : "v"(rflag + ((f >> 4) & 1u) * NSTEP + (f >> 5)), "v"(1u) : "memory");
            } else if (lane == 0) asm volatile("global_atomic_add %0, %1, off" : : "v"(cntp + (f >> 4) * 32u), "v"(1u) : "memory");
        };
        auto round = [&](f32x4& c0, f32x4& c1, f32x4& n0, f32x4& n1) {
            f16x8 h;
#pragma unroll
            for (int e = 0; e < 4; ++e) { h[e] = (_Float16)c0[e]; h[4 + e] = (_Float16)c1[e]; }
            asm volatile("" : "+v"(h) : : "memory");
            unsigned t = 0;
            const bool draw = f2 < flim;
            {
                unsigned long long save;
                const unsigned msk = __builtin_amdgcn_readfirstlane(draw ? 1u : 0u);
                asm volatile("s_mov_b64 %1, exec\n\ts_mov_b32 exec_lo, %4\n\ts_mov_b32 exec_hi, 0\n\tglobal_atomic_add %0, %2, %3, off sc0\n\ts_mov_b64 exec, %1"
                             : "+v"(t), "=&s"(save) : "v"(tick), "v"(1u), "s"(msk) : "memory");
            }
            const unsigned off = ((((f0 >> 4) & 1u) * NSTEP + (f0 >> 5)) * 16u + (f0 & 15u)) * 1024u + (unsigned)lane * 16u;
#pragma unroll
            for (int q = 0; q < NR - 1; ++q) {
                st16_sys(dst[q], off, h);
            }
            ld(c0, c1, f2);
            asm volatile("s_waitcnt vmcnt(9)" : "+v"(t), "+v"(n0), "+v"(n1) : : "memory");
            if (fp != 0xffffffffu) count(fp);
            fp = f0; f0 = f1; f1 = f2;
            f2 = draw ? __builtin_amdgcn_readfirstlane(t) - tbase : 0xffffffffu;
        };
        while (true) {
            round(a0, a1, b0, b1);
            if (f0 >= (unsigned)NFRAG) break;
            round(b0, b1, a0, a1);
            if (f0 >= (unsigned)NFRAG) break;
        }
        asm volatile("s_waitcnt vmcnt(0)" : "+v"(a0), "+v"(a1), "+v"(b0), "+v"(b1) : : "memory");
        count(fp);
        STAMP(17 + wave);
        return;
    }

    const unsigned ctl = lds_base + CTL_OFF;
    const int xcd = blk & 7;
    const int t = (xcd >> 1) * 30 + ((blk >> 3) - 2);
    const int im = t >> 3;
    const int jn = (xcd & 1) * 8 + (t & 7);
    const int src = (rank + ((im + 1) >> 1)) & (NR - 1);
    const int half = (im == 0) ? 0 : ((im + 1) & 1);
    const int ef0 = (im == 0) ? 14 : im - 1;
    const float* wl = p.w + (size_t)(4 * fq) * NTW + (size_t)rank * NS + 4 * (4 * (fr16 & 3) + (fr16 >> 2));
    const char* wdst = uni_ptr(myws + W16_OFF);
    unsigned* fwp = (unsigned*)(myws + FLAGW_OFF);
    const int li = (blk >> 3) - 2;

    const int wm_ = wave & 3, wn_ = (wave >> 2) & 1;
    const int ek = wm_ >> 1;
    const bool has_e = wave >= NCONS || (((wm_ & 1) == wn_) && (ek == 0 || im == 0));
    f32x4 acc[4][4], ex[4];

    if (wave < NCONS) {
        STAMP(0);
        const int cst = im + NMT * (wave >> 1), cj = jn, cp = wave & 1;
        f32x4 wv[8];
        if (wave < CV_WAVES) {
            const float* wu = wl + (size_t)(BK * cst) * NTW + cj * BN + 64 * cp;
#pragma unroll
            for (int e = 0; e < 8; ++e) wv[e] = ld_w(wu + (size_t)((e >> 2) * 16 + (e & 3)) * NTW);
        }
        asm volatile("" ::: "memory");
        __builtin_amdgcn_s_barrier();
        asm volatile("" ::: "memory");
        const unsigned ep = __builtin_amdgcn_readfirstlane(lds_ld4(ctl + 16).x);
        if (wave < CV_WAVES) {
            f16x8 h[4];
#pragma unroll
            for (int i = 0; i < 4; ++i)
#pragma unroll
                for (int e = 0; e < 8; ++e) h[i][e] = (_Float16)wv[e][i];
            asm volatile("" : "+v"(h[0]), "+v"(h[1]), "+v"(h[2]), "+v"(h[3]) : : "memory");
#pragma unroll
            for (int i = 0; i < 4; ++i) st16_dev(wdst, (unsigned)((cj * NSTEP + cst) * B_BYTES + cp * 4096 + i * 1024 + lane * 16), h[i]);
            wait_vm<0>();
            if (lane == 0) __hip_atomic_store(fwp + (cj * 2 + cp) * NSTEP + cst, ep, __ATOMIC_RELAXED, __HIP_MEMORY_SCOPE_AGENT);
        }
        STAMP(1);
#pragma unroll
        for (int j = 0; j < 4; ++j) {
#pragma unroll
            for (int i = 0; i < 4; ++i) acc[i][j] = (f32x4){0.f, 0.f, 0.f, 0.f};
            ex[j] = (f32x4){0.f, 0.f, 0.f, 0.f};
        }
        const LAS char* abase = (const LAS char*)lds + wm_ * 4096 + lane * 16;
        const LAS char* ebase = (const LAS char*)lds + A_BYTES + ek * 1024 + lane * 16;
        const LAS char* bbase = (const LAS char*)lds + A_BYTES + E_BYTES + wn_ * 4096 + lane * 16;
        const LAS u32x4* ctlp = (const LAS u32x4*)((const LAS char*)lds + CTL_OFF);
        const unsigned done_addr = ctl + 64 + wave * 4;
        ACC_DECL(a_wa); ACC_DECL(a_tot);
        auto spin_avail = [&](int s) {
            ACC_BEG(a_wa);
            while (true) {
                const u32x4 f = lds_ld4(ctl);
                if ((int)__builtin_amdgcn_readfirstlane(min(min(f.x, f.y), f.z)) > s) break;
                __builtin_amdgcn_s_sleep(1);
            }
            if (s > 0) ACC_END(a_wa);
        };
        auto run = [&](auto he) {
            constexpr bool HE = decltype(he)::value;
            Frags F0, F1;
            u32x4 cw_;
            auto load = [&](Frags& F, int slot) {
                const int so = slot * SLOT_BYTES;
#pragma unroll
                for (int j = 0; j < 4; ++j) F.b[j] = *(const LAS f16x8*)(bbase + so + j * 1024);
#pragma unroll
                for (int i = 0; i < 4; ++i) F.a[i] = *(const LAS f16x8*)(abase + so + i * 1024);
                if (HE) F.e = *(const LAS f16x8*)(ebase + so);
                cw_ = *(const volatile LAS u32x4*)ctlp;
            };
            auto mma = [&](const Frags& F) {
#pragma unroll
                for (int i = 0; i < 4; ++i)
#pragma unroll
                    for (int j = 0; j < 4; ++j)
                        acc[i][j] = __builtin_amdgcn_mfma_f32_16x16x32_f16(F.b[j], F.a[i], acc[i][j], 0, 0, 0);
                if (HE) {
#pragma unroll
                    for (int j = 0; j < 4; ++j) ex[j] = __builtin_amdgcn_mfma_f32_16x16x32_f16(F.b[j], F.e, ex[j], 0, 0, 0);
                }
            };
            auto step = [&](Frags& Fc, Frags& Fn, int s, int nslot) {
                __builtin_amdgcn_s_waitcnt(0xC07F);
                lds_st(done_addr, (unsigned)(s + 1));
                const int av = (int)__builtin_amdgcn_readfirstlane(min(min(cw_.x, cw_.y), cw_.z));
                if (av <= s + 1 && s + 1 < NSTEP) spin_avail(s + 1);
                __builtin_amdgcn_sched_barrier(0);
                load(Fn, nslot);
                mma(Fc);
#pragma unroll
                for (int g = 0; g < (HE ? 10 : 9); ++g) {
                    __builtin_amdgcn_sched_group_barrier(0x008, 1, 0);
                    __builtin_amdgcn_sched_group_barrier(0x100, 1, 0);
                }
                __builtin_amdgcn_sched_barrier(0);
            };
            spin_avail(0);
            load(F0, 0);
            if (wave == 0) STAMP(2);
            ACC_BEG(a_tot);
            int slot = 0;
            for (int s = 0; s < NSTEP; s += 2) {
                step(F0, F1, s, slot + 1);
                slot = (slot == NSLOT - 2) ? 0 : slot + 2;
                step(F1, F0, s + 1, slot);
            }
            ACC_END(a_tot);
        };
        if (has_e) run(std::integral_constant<bool, true>{}); else run(std::integral_constant<bool, false>{});
        if (wave == 0) STAMP(3);
        if (wave == 0) { ACC_OUT(a_wa, 28); ACC_OUT(a_tot, 30); }
        if (__builtin_amdgcn_readfirstlane(lds_ld4(ctl).w) != 0u) {
            const float nanv = __builtin_nanf("");
#pragma unroll
            for (int j = 0; j < 4; ++j) {
#pragma unroll
                for (int i = 0; i < 4; ++i) acc[i][j] = (f32x4){nanv, nanv, nanv, nanv};
                ex[j] = (f32x4){nanv, nanv, nanv, nanv};
            }
        }
    } else {
        __builtin_amdgcn_s_setprio(LOADER_PRIO);
        unsigned ep;
        if (wave == NCONS) {
            ep = __builtin_amdgcn_readfirstlane(__hip_atomic_load(epw, __ATOMIC_RELAXED, __HIP_MEMORY_SCOPE_AGENT)) + 1u;
            if (lane < 32) lds_st(ctl + lane * 4, lane == 4 ? ep : 0u);
            if (lane == 0) __hip_atomic_store(epw, ep, __ATOMIC_RELAXED, __HIP_MEMORY_SCOPE_AGENT);
            asm volatile("s_waitcnt lgkmcnt(0)" ::: "memory");
        }
        asm volatile("" ::: "memory");
        __builtin_amdgcn_s_barrier();
        asm volatile("" ::: "memory");
        ep = __builtin_amdgcn_readfirstlane(lds_ld4(ctl + 16).x);
        unsigned* tmo = (unsigned*)(myws + TMO_OFF);
        if (wave == NCONS + 3) {
            if (im == 0) { watcher(p, myws, rank, lane, ep, jn, 16); STAMP(15); return; }
            if (im == 0) return;
            {
                const char* ldst = uni_ptr(myws + XS_OFF + ((size_t)(ep & 1u) * NR + rank) * XS_SLOT);
                unsigned* lflag = (unsigned*)(myws + FLAGX_OFF) + rank * 2 * NSTEP;
                for (int o = (im - 1) * 16 + jn; o < 2 * NSTEP; o += 14 * 16) {
                    const int unit = (o & 1) * NSTEP + (o >> 1);
                    const float* xu = p.x + (size_t)((o & 1) * BM + fr16) * KDX + (o >> 1) * BK + 4 * fq;
#pragma unroll
                    for (int hh = 0; hh < 2; ++hh) {
                        f32x4 xv[8][2];
#pragma unroll
                        for (int m = 0; m < 8; ++m) {
                            xv[m][0] = ld_x(xu + (size_t)((hh * 8 + m) * 16) * KDX);
                            xv[m][1] = ld_x(xu + (size_t)((hh * 8 + m) * 16) * KDX + 16);
                        }
                        f16x8 h[8];
#pragma unroll
                        for (int m = 0; m < 8; ++m)
#pragma unroll
                            for (int e = 0; e < 4; ++e) { h[m][e] = (_Float16)xv[m][0][e]; h[m][4 + e] = (_Float16)xv[m][1][e]; }
                        asm volatile("" : "+v"(h[0]), "+v"(h[1]), "+v"(h[2]), "+v"(h[3]), "+v"(h[4]), "+v"(h[5]), "+v"(h[6]), "+v"(h[7]) : : "memory");
#pragma unroll
                        for (int m = 0; m < 8; ++m) st16_dev(ldst, (unsigned)((unit * 16 + hh * 8 + m) * 1024 + lane * 16), h[m]);
                    }
                    wait_vm<0>();
                    if (lane == 0) __hip_atomic_store(lflag + unit, ep, __ATOMIC_RELAXED, __HIP_MEMORY_SCOPE_AGENT);
                }
            }
            STAMP(5);
            for (int st = NMT * CV_WAVES / 2 + im - 1; st < NSTEP; st += NMT - 1) {
                const int hj = jn;
                f32x4 wv[2][8];
#pragma unroll
                for (int pp = 0; pp < 2; ++pp)
#pragma unroll
                    for (int e = 0; e < 8; ++e)
                        wv[pp][e] = ld_w(wl + (size_t)(BK * st + (e >> 2) * 16 + (e & 3)) * NTW + hj * BN + 64 * pp);
                f16x8 h[8];
#pragma unroll
                for (int pp = 0; pp < 2; ++pp)
#pragma unroll
                    for (int i = 0; i < 4; ++i)
#pragma unroll
                        for (int e = 0; e < 8; ++e) h[4 * pp + i][e] = (_Float16)wv[pp][e][i];
                asm volatile("" : "+v"(h[0]), "+v"(h[1]), "+v"(h[2]), "+v"(h[3]), "+v"(h[4]), "+v"(h[5]), "+v"(h[6]), "+v"(h[7]) : : "memory");
#pragma unroll
                for (int i = 0; i < 8; ++i) st16_dev(wdst, (unsigned)((hj * NSTEP + st) * B_BYTES + i * 1024 + lane * 16), h[i]);
                wait_vm<0>();
                if (lane < 2) __hip_atomic_store(fwp + (hj * 2 + lane) * NSTEP + st, ep, __ATOMIC_RELAXED, __HIP_MEMORY_SCOPE_AGENT);
            }
            STAMP(14);
            if (!EP_WARM) return;
#pragma unroll
            for (int j = 0; j < 4; ++j) {
#pragma unroll
                for (int i = 0; i < 4; ++i) acc[i][j] = (f32x4){0.f, 0.f, 0.f, 0.f};
                ex[j] = (f32x4){0.f, 0.f, 0.f, 0.f};
            }
        } else {
        if (wave == NCONS) {
            const char* ws16 = myws + W16_OFF + (size_t)jn * NSTEP * B_BYTES;
            const unsigned* fw = (const unsigned*)(myws + FLAGW_OFF) + jn * 2 * NSTEP;
            loader<false, false>(ws16, ws16, B_BYTES, lds_base, A_BYTES + E_BYTES, 0, ctl + 8, fw, fw + NSTEP, ep, tmo, lane, myws, 8);
        } else {
            const int la = wave - NCONS - 1;
            const char* xsb = myws + XS_OFF + (size_t)(ep & 1u) * NR * XS_SLOT;
            const char* xs = xsb + (size_t)src * XS_SLOT + (size_t)half * NSTEP * A_BYTES + la * 8192;
            const char* xe = xsb + (size_t)rank * XS_SLOT + (size_t)NSTEP * A_BYTES + (ef0 + la) * 1024;
            const unsigned* fx = (const unsigned*)(myws + FLAGX_OFF);
            const unsigned* f0 = fx + (src * 2 + half) * NSTEP;
            if (la == 0 || im == 0)
                loader<true, true>(xs, xe, A_BYTES, lds_base, la * 8192, A_BYTES + la * 1024, ctl + 4 * la, f0, fx + (rank * 2 + 1) * NSTEP, ep, tmo, lane, myws, 10 + 2 * la, im != 0, ONE_POLLER ? 1 + la : 0);
            else
                loader<true, false>(xs, xs, A_BYTES, lds_base, la * 8192, 0, ctl + 4 * la, f0, f0, ep, tmo, lane, myws, 10 + 2 * la, true, ONE_POLLER ? 2 : 0);
        }
        return;
        }
    }
    {
        const unsigned live = __builtin_amdgcn_readfirstlane(wave < NCONS ? 0xffffffffu : 0u);
        float* oc = p.out + jn * BN + wn_ * 64 + 4 * fq;
        float* op = oc + (size_t)(src * MS + half * BM + wm_ * 64 + fr16) * NS;
        float* oe = oc + (size_t)(rank * MS + BM + ((ef0 + ek) & 15) * 16 + fr16) * NS;
        auto blockout = [&](const f32x4 (&y)[4], float* dst) {
            f32x4 g[4];
#pragma unroll
            for (int j = 0; j < 4; ++j) g[j] = sigm4(y[j]);
#pragma unroll
            for (int r = 0; r < 4; ++r) {
                f32x4 v;
#pragma unroll
                for (int j = 0; j < 4; ++j) { float t = y[j][r] * g[j][r]; asm volatile("" : "+v"(t)); v[j] = t; }
                st_out(dst + 16 * r, v, live);
            }
        };
#pragma unroll
        for (int i = 0; i < 4; ++i) blockout(acc[i], op + (size_t)(16 * i) * NS);
        if (has_e) blockout(ex, oe);
    }
}

}

extern "C" void kernel_launch(int rank, int nranks, ncclComm_t comm,
                              hipStream_t stream,
                              void* const* d_in, const int* in_sizes, int n_in,
                              void* d_out, int out_size,
                              const int* in_dtypes, int out_dtype,
                              void* const* d_out_all,
                              void* const* d_ws_all, size_t ws_bytes) {
    (void)comm; (void)in_dtypes; (void)out_dtype; (void)d_out_all;
    if (nranks != NR || (unsigned)rank >= (unsigned)NR || n_in < 2 || ws_bytes < WS_NEED ||
        in_sizes[0] != MS * KD || in_sizes[1] != KD * NT || out_size != MT * NS) {
        fprintf(stderr, "kernel_launch: unexpected problem shape\n");
        return;
    }
    Params p;
    p.x = (const float*)d_in[0];
    p.w = (const float*)d_in[1];
    p.out = (float*)d_out;
    for (int i = 0; i < NR; ++i) p.ws[i] = (char*)d_ws_all[i];
    p.rank = rank;
    hipLaunchKernelGGL(fused_kernel, dim3(NBLK), dim3(NTHREADS), 0, stream, p);
    HIP_CHECK(hipGetLastError());
}
```

```cpp
#include <hip/hip_runtime.h>
#include <rccl/rccl.h>
#include <cstdio>
#include <cstdint>
#include <type_traits>

#define HIP_CHECK(expr) do { hipError_t _e = (expr); if (_e != hipSuccess) { \
    fprintf(stderr, "HIP error %s at %s:%d\n", hipGetErrorString(_e), __FILE__, __LINE__); \
    return; } } while (0)

namespace {

typedef float f32x4 __attribute__((ext_vector_type(4)));
typedef _Float16 f16x8 __attribute__((ext_vector_type(8)));
typedef unsigned u32x4 __attribute__((ext_vector_type(4)));
#define LAS __attribute__((address_space(3)))

constexpr int NR = 8;
constexpr int MT = 4096, KD = 4096, NT = 16384;
constexpr int MS = MT / NR;
constexpr int NS = NT / NR;
constexpr int BM = 256, BN = 128, BK = 32;
constexpr int NSTEP = KD / BK;
constexpr int NSLOT = 6;
constexpr int A_BYTES = BM * BK * 2;
constexpr int E_BYTES = 2048;
constexpr int B_BYTES = BN * BK * 2;
constexpr int SLOT_BYTES = A_BYTES + E_BYTES + B_BYTES;
constexpr int CTL_OFF = NSLOT * SLOT_BYTES;
constexpr int LDS_BYTES = CTL_OFF + 128;
constexpr int NCONS = 8;
constexpr int NWAVES = NCONS + 4;
constexpr int NTHREADS = NWAVES * 64;
constexpr int NBLK = 256;
constexpr int NCB = 16;
constexpr int NMT = 15;
constexpr int CWPB = 4;
constexpr int CWPB_KEEP = 3;
constexpr int RETIRE_F = 3584;
constexpr int NCW = NCB * CWPB;
constexpr int NFRAG = (MS / 16) * NSTEP;
constexpr int LDEPTH = 3;
constexpr int CV_WAVES = 2;

constexpr size_t EPOCH_OFF = 0;
constexpr size_t TMO_OFF = 16384;
constexpr size_t TICK_OFF = 16384 + 256;
constexpr size_t CNT_OFF = 32768;
constexpr size_t FLAGW_OFF = 65536;
constexpr size_t FLAGX_OFF = 65536 + 16384;
constexpr size_t XS_OFF = 1u << 20;
constexpr size_t XS_SLOT = (size_t)MS * KD * 2;
constexpr size_t W16_OFF = XS_OFF + 2 * NR * XS_SLOT;
constexpr size_t WS_NEED = W16_OFF + (size_t)KD * NS * 2;
constexpr unsigned long long DEADLINE_TICKS = 100000000ull;

struct Params {
    const float* x;
    const float* w;
    float* out;
    char* ws[NR];
    int rank;
};

__device__ __forceinline__ unsigned long long rt() { return __builtin_amdgcn_s_memrealtime(); }
__device__ __forceinline__ void lds_st(unsigned addr, unsigned v) {
    asm volatile("ds_write_b32 %0, %1" : : "v"(addr), "v"(v) : "memory");
}
__device__ __forceinline__ u32x4 lds_ld4(unsigned addr) {
    u32x4 v;
    asm volatile("ds_read_b128 %0, %1\n\ts_waitcnt lgkmcnt(0)" : "=&v"(v) : "v"(addr) : "memory");
    return v;
}
__device__ __forceinline__ void glds4k(const void* gsrc, unsigned lds_dst) {
    unsigned keep;
    asm volatile("s_mov_b32 %0, m0\n\ts_mov_b32 m0, %2\n\ts_nop 0\n\t"
                 "global_load_lds_dwordx4 %1, off sc1\n\t"
                 "global_load_lds_dwordx4 %1, off offset:1024 sc1\n\t"
                 "global_load_lds_dwordx4 %1, off offset:2048 sc1\n\t"
                 "global_load_lds_dwordx4 %1, off offset:3072 sc1\n\t"
                 "s_mov_b32 m0, %0"
                 : "=&s"(keep) : "v"(gsrc), "s"(lds_dst) : "memory");
}
__device__ __forceinline__ void glds1k(const void* gsrc, unsigned lds_dst) {
    unsigned keep;
    asm volatile("s_mov_b32 %0, m0\n\ts_mov_b32 m0, %2\n\ts_nop 0\n\t"
                 "global_load_lds_dwordx4 %1, off sc1\n\t"
                 "s_mov_b32 m0, %0"
                 : "=&s"(keep) : "v"(gsrc), "s"(lds_dst) : "memory");
}
template <int N> __device__ __forceinline__ void wait_vm() {
    asm volatile("s_waitcnt vmcnt(%0)" : : "i"(N) : "memory");
}
__device__ __forceinline__ void st16_sys(const char* base, unsigned off, f16x8 v) {
    asm volatile("global_store_dwordx4 %0, %1, %2 sc0 sc1\n\ts_nop 1" : : "v"(off), "v"(v), "s"(base) : "memory");
}
__device__ __forceinline__ void st16_dev(const char* base, unsigned off, f16x8 v) {
    asm volatile("global_store_dwordx4 %0, %1, %2 sc1\n\ts_nop 1" : : "v"(off), "v"(v), "s"(base) : "memory");
}
__device__ __forceinline__ void st_out(float* p, f32x4 v) {
    asm volatile("global_store_dwordx4 %0, %1, off sc1\n\ts_nop 1" : : "v"(p), "v"(v) : "memory");
}
__device__ __forceinline__ const char* uni_ptr(const char* p) {
    const unsigned long long u = (unsigned long long)p;
    const unsigned lo = __builtin_amdgcn_readfirstlane((unsigned)u), hi = __builtin_amdgcn_readfirstlane((unsigned)(u >> 32));
    return (const char*)(((unsigned long long)hi << 32) | lo);
}
__device__ __forceinline__ f32x4 ld_w(const float* p) { return __builtin_nontemporal_load((const f32x4*)p); }
__device__ __forceinline__ f32x4 sigm4(f32x4 y) {
    f32x4 c, e, r;
#pragma unroll
    for (int i = 0; i < 4; ++i) c[i] = __builtin_amdgcn_fmed3f(y[i], -60.0f, 60.0f);
    c = c * -1.44269504088896340736f;
#pragma unroll
    for (int i = 0; i < 4; ++i) e[i] = __builtin_amdgcn_exp2f(c[i]);
    e = e + 1.0f;
#pragma unroll
    for (int i = 0; i < 4; ++i) r[i] = __builtin_amdgcn_rcpf(e[i]);
    return r;
}

template <bool ISX, bool EXTRA>
__device__ __forceinline__ void loader(const char* src, const char* esrc, unsigned stride, unsigned lds_base,
                                       unsigned lds_off, unsigned elds_off, unsigned full_addr,
                                       const unsigned* flag0, const unsigned* flag1,
                                       unsigned ep, unsigned* tmo, int lane, int role) {
    constexpr int PIECES = 8 + (EXTRA ? 1 : 0);
    static_assert(PIECES * LDEPTH <= 60 && LDEPTH == 3, "vmcnt is 6 bits; retire ladder below assumes <= 2 unpublished");
    const unsigned ctl = lds_base + CTL_OFF;
    const char* lsrc = src + lane * 16;
    const char* le = esrc + lane * 16;
    const unsigned* pollp = (lane < 32 ? flag0 : flag1) + (lane & 31);
    int pub = 0, wm = 0, fr = NSLOT;
    int slot = 0;
    for (int s = 0; s < NSTEP; ++s) {
        if (s >= wm && role == 2) {
            while (true) {
                unsigned w;
                asm volatile("ds_read_b32 %0, %1 offset:24" : "=&v"(w) : "v"(ctl) : "memory");
                const bool had = pub < s;
                if (had) { wait_vm<0>(); lds_st(full_addr, (unsigned)s); pub = s; }
                asm volatile("s_waitcnt lgkmcnt(0)" : "+v"(w) : : "memory");
                wm = (int)__builtin_amdgcn_readfirstlane(w);
                if (wm > s) break;
                if (!had) __builtin_amdgcn_s_sleep(1);
            }
        }
        if (s >= wm) {
            const unsigned long long t0 = rt();
            const unsigned* pp = pollp + (s & ~31);
            unsigned spins = 0;
            while (true) {
                unsigned fl;
                if (ISX) asm volatile("global_load_dword %0, %1, off sc0 sc1" : "=&v"(fl) : "v"(pp) : "memory");
                else     asm volatile("global_load_dword %0, %1, off sc1" : "=&v"(fl) : "v"(pp) : "memory");
                if (pub < s) { wait_vm<1>(); lds_st(full_addr, (unsigned)s); pub = s; }
                asm volatile("s_waitcnt vmcnt(0)" : "+v"(fl) : : "memory");
                const unsigned long long m = __ballot((int)(fl - ep) >= 0);
                const unsigned both = (unsigned)m & (unsigned)(m >> 32);
                wm = (s & ~31) + (both == 0xffffffffu ? 32 : __builtin_ctz(~both));
                if (wm > s) break;
                if ((++spins & 63u) == 0u) {
                    const unsigned tv = __builtin_amdgcn_readfirstlane(__hip_atomic_load(tmo, __ATOMIC_RELAXED, __HIP_MEMORY_SCOPE_AGENT));
                    if (tv == ep || rt() - t0 > DEADLINE_TICKS) {
                        if (lane == 0) __hip_atomic_store(tmo, ep, __ATOMIC_RELAXED, __HIP_MEMORY_SCOPE_AGENT);
                        lds_st(ctl + 12, 1u);
                        wait_vm<0>();
                        wm = NSTEP;
                        break;
                    }
                }
                if (s < NSTEP - 12) __builtin_amdgcn_s_sleep(32); else __builtin_amdgcn_s_sleep(2);
            }
            if (role == 1) lds_st(ctl + 24, (unsigned)wm);
        }
        if (s >= fr) {
            while (true) {
                const u32x4 d0 = lds_ld4(ctl + 64), d1 = lds_ld4(ctl + 80);
                unsigned mn = min(min(min(d0.x, d0.y), min(d0.z, d0.w)), min(min(d1.x, d1.y), min(d1.z, d1.w)));
                fr = (int)__builtin_amdgcn_readfirstlane(mn) + NSLOT;
                if (fr > s) break;
                if (pub < s) {
                    if (s - pub == 1) wait_vm<0>(); else wait_vm<PIECES>();
                    ++pub; lds_st(full_addr, (unsigned)pub);
                } else {
                    __builtin_amdgcn_s_sleep(1);
                }
            }
        }
        const unsigned dst = lds_base + (unsigned)slot * SLOT_BYTES;
        const size_t go = (size_t)s * stride;
        glds4k(lsrc + go, dst + lds_off);
        glds4k(lsrc + go + 4096, dst + lds_off + 4096);
        if (EXTRA) glds1k(le + go, dst + elds_off);
        slot = (slot == NSLOT - 1) ? 0 : slot + 1;
        if (s + 1 - pub >= LDEPTH) { wait_vm<PIECES * (LDEPTH - 1)>(); ++pub; lds_st(full_addr, (unsigned)pub); }
    }
    wait_vm<0>();
    lds_st(full_addr, (unsigned)NSTEP);
}

__device__ __forceinline__ void watcher(const Params& p, char* myws, int rank, int lane, unsigned ep, int w) {
    unsigned* tmo = (unsigned*)(myws + TMO_OFF);
    const unsigned* cntp = (const unsigned*)(myws + CNT_OFF);
    unsigned* flagp = (unsigned*)(p.ws[(rank + 1 + lane) & (NR - 1)] + FLAGX_OFF) + rank * 2 * NSTEP;
    const unsigned need = ep * 16u;
    const unsigned long long t0 = rt();
    unsigned spins = 0;
    for (unsigned o = w; o < 2u * NSTEP; o += 16) {
        while (true) {
            unsigned c;
            asm volatile("global_load_dword %0, %1, off sc1\n\ts_waitcnt vmcnt(0)" : "=&v"(c) : "v"(cntp + o * 32u) : "memory");
            if ((int)(__builtin_amdgcn_readfirstlane(c) - need) >= 0) break;
            if ((++spins & 255u) == 0u && rt() - t0 > DEADLINE_TICKS) {
                if (lane == 0) __hip_atomic_store(tmo, ep, __ATOMIC_RELAXED, __HIP_MEMORY_SCOPE_AGENT);
                return;
            }
            __builtin_amdgcn_s_sleep(1);
        }
        if (lane < NR - 1)
            asm volatile("global_store_dword %0, %1, off sc0 sc1" : : "v"(flagp + (o & 1u) * NSTEP + (o >> 1)), "v"(ep) : "memory");
    }
}

struct Frags { f16x8 a[4], b[4], e; };

__global__ void __launch_bounds__(NTHREADS, 3) fused_kernel(Params p) {
    __shared__ __attribute__((aligned(1024))) char lds[LDS_BYTES];
    const int tid = threadIdx.x;
    const int lane = tid & 63;
    const int wave = __builtin_amdgcn_readfirstlane(tid >> 6);
    const int blk = blockIdx.x;
    const int rank = p.rank & (NR - 1);
    char* const myws = p.ws[rank];
    const int fr16 = lane & 15, fq = lane >> 4;
    unsigned* const epw = (unsigned*)(myws + EPOCH_OFF) + blk * 16;
    const unsigned lds_base = (unsigned)(size_t)(LAS char*)lds;

    if (blk < NCB) {
        if (wave >= CWPB) return;
        const int cw = wave * NCB + blk;
        const float* xl = p.x + (size_t)fr16 * KD + 4 * fq;
        auto ld = [&](f32x4& a, f32x4& b, unsigned f) {
            f = min(f, (unsigned)NFRAG - 1u);
            const float* xp = xl + (size_t)((f & 31u) * 16u) * KD + (f >> 5) * BK;
            asm volatile("global_load_dwordx4 %0, %2, off\n\tglobal_load_dwordx4 %1, %2, off offset:64" : "=&v"(a), "=&v"(b) : "v"(xp) : "memory");
        };
        f32x4 a0, a1, b0, b1;
        ld(a0, a1, (unsigned)cw);
        ld(b0, b1, (unsigned)(cw + NCW));
        const unsigned ep = __builtin_amdgcn_readfirstlane(__hip_atomic_load(epw + wave, __ATOMIC_RELAXED, __HIP_MEMORY_SCOPE_AGENT)) + 1u;
        asm volatile("s_waitcnt vmcnt(0)" : "+v"(a0), "+v"(a1), "+v"(b0), "+v"(b1) : : "memory");
        if (lane == 0) __hip_atomic_store(epw + wave, ep, __ATOMIC_RELAXED, __HIP_MEMORY_SCOPE_AGENT);
        const size_t slot_off = XS_OFF + ((size_t)(ep & 1u) * NR + rank) * XS_SLOT;
        const char* dst[NR - 1];
#pragma unroll
        for (int q = 0; q < NR - 1; ++q) {
            const unsigned pr = (unsigned)(rank + 1 + (q + cw) % (NR - 1)) & (NR - 1);
            dst[q] = uni_ptr(p.ws[pr] + slot_off);
        }
        unsigned* tick = (unsigned*)(myws + TICK_OFF);
        unsigned* cntp = (unsigned*)(myws + CNT_OFF);
        const unsigned tbase = (ep - 1u) * (unsigned)(NFRAG - 3 * NCW + NCB * CWPB_KEEP) - 3u * (unsigned)NCW;
        const unsigned flim = wave < CWPB_KEEP ? (unsigned)NFRAG : (unsigned)RETIRE_F;
        unsigned f0 = (unsigned)cw, f1 = f0 + NCW, f2 = f1 + NCW, fp = 0xffffffffu;
        auto count = [&](unsigned f) {
            if (lane == 0) asm volatile("global_atomic_add %0, %1, off" : : "v"(cntp + (f >> 4) * 32u), "v"(1u) : "memory");
        };
        auto round = [&](f32x4& c0, f32x4& c1, f32x4& n0, f32x4& n1) {
            f16x8 h;
#pragma unroll
            for (int e = 0; e < 4; ++e) { h[e] = (_Float16)c0[e]; h[4 + e] = (_Float16)c1[e]; }
            asm volatile("" : "+v"(h) : : "memory");
            unsigned t = 0;
            const bool draw = f2 < flim;
            {
                unsigned long long save;
                const unsigned msk = __builtin_amdgcn_readfirstlane(draw ? 1u : 0u);
                asm volatile("s_mov_b64 %1, exec\n\ts_mov_b32 exec_lo, %4\n\ts_mov_b32 exec_hi, 0\n\tglobal_atomic_add %0, %2, %3, off sc0\n\ts_mov_b64 exec, %1"
                             : "+v"(t), "=&s"(save) : "v"(tick), "v"(1u), "s"(msk) : "memory");
            }
            const unsigned off = ((((f0 >> 4) & 1u) * NSTEP + (f0 >> 5)) * 16u + (f0 & 15u)) * 1024u + (unsigned)lane * 16u;
#pragma unroll
            for (int q = 0; q < NR - 1; ++q) st16_sys(dst[q], off, h);
            ld(c0, c1, f2);
            asm volatile("s_waitcnt vmcnt(9)" : "+v"(t), "+v"(n0), "+v"(n1) : : "memory");
            if (fp != 0xffffffffu) count(fp);
            fp = f0; f0 = f1; f1 = f2;
            f2 = draw ? __builtin_amdgcn_readfirstlane(t) - tbase : 0xffffffffu;
        };
        while (true) {
            round(a0, a1, b0, b1);
            if (f0 >= (unsigned)NFRAG) break;
            round(b0, b1, a0, a1);
            if (f0 >= (unsigned)NFRAG) break;
        }
        asm volatile("s_waitcnt vmcnt(0)" : "+v"(a0), "+v"(a1), "+v"(b0), "+v"(b1) : : "memory");
        count(fp);
        return;
    }

    const unsigned ctl = lds_base + CTL_OFF;
    const int xcd = blk & 7;
    const int t = (xcd >> 1) * 30 + ((blk >> 3) - 2);
    const int im = t >> 3;
    const int jn = (xcd & 1) * 8 + (t & 7);
    const int src = (rank + ((im + 1) >> 1)) & (NR - 1);
    const int half = (im == 0) ? 0 : ((im + 1) & 1);
    const int ef0 = (im == 0) ? 14 : im - 1;
    const float* wl = p.w + (size_t)(4 * fq) * NT + (size_t)rank * NS + jn * BN + 4 * (4 * (fr16 & 3) + (fr16 >> 2));
    const char* wdst = uni_ptr(myws + W16_OFF + (size_t)jn * NSTEP * B_BYTES);
    unsigned* fwp = (unsigned*)(myws + FLAGW_OFF) + jn * 2 * NSTEP;

    if (wave < NCONS) {
        const int cst = im + NMT * (wave >> 1), cp = wave & 1;
        f32x4 wv[8];
        if (wave < CV_WAVES) {
            const float* wu = wl + (size_t)(BK * cst) * NT + 64 * cp;
#pragma unroll
            for (int e = 0; e < 8; ++e) wv[e] = ld_w(wu + (size_t)((e >> 2) * 16 + (e & 3)) * NT);
        }
        asm volatile("" ::: "memory");
        __builtin_amdgcn_s_barrier();
        asm volatile("" ::: "memory");
        const unsigned ep = __builtin_amdgcn_readfirstlane(lds_ld4(ctl + 16).x);
        if (wave < CV_WAVES) {
            f16x8 h[4];
#pragma unroll
            for (int i = 0; i < 4; ++i)
#pragma unroll
                for (int e = 0; e < 8; ++e) h[i][e] = (_Float16)wv[e][i];
            asm volatile("" : "+v"(h[0]), "+v"(h[1]), "+v"(h[2]), "+v"(h[3]) : : "memory");
#pragma unroll
            for (int i = 0; i < 4; ++i) st16_dev(wdst, (unsigned)(cst * B_BYTES + cp * 4096 + i * 1024 + lane * 16), h[i]);
            wait_vm<0>();
            if (lane == 0) __hip_atomic_store(fwp + cp * NSTEP + cst, ep, __ATOMIC_RELAXED, __HIP_MEMORY_SCOPE_AGENT);
        }
        const int wm_ = wave & 3, wn_ = wave >> 2;
        const int ek = wm_ >> 1;
        const bool has_e = ((wm_ & 1) == wn_) && (ek == 0 || im == 0);
        f32x4 acc[4][4], ex[4];
#pragma unroll
        for (int j = 0; j < 4; ++j) {
#pragma unroll
            for (int i = 0; i < 4; ++i) acc[i][j] = (f32x4){0.f, 0.f, 0.f, 0.f};
            ex[j] = (f32x4){0.f, 0.f, 0.f, 0.f};
        }
        const LAS char* abase = (const LAS char*)lds + wm_ * 4096 + lane * 16;
        const LAS char* ebase = (const LAS char*)lds + A_BYTES + ek * 1024 + lane * 16;
        const LAS char* bbase = (const LAS char*)lds + A_BYTES + E_BYTES + wn_ * 4096 + lane * 16;
        const LAS u32x4* ctlp = (const LAS u32x4*)((const LAS char*)lds + CTL_OFF);
        const unsigned done_addr = ctl + 64 + wave * 4;
        auto spin_avail = [&](int s) {
            while (true) {
                const u32x4 f = lds_ld4(ctl);
                if ((int)__builtin_amdgcn_readfirstlane(min(min(f.x, f.y), f.z)) > s) break;
                __builtin_amdgcn_s_sleep(1);
            }
        };
        auto run = [&](auto he) {
            constexpr bool HE = decltype(he)::value;
            Frags F0, F1;
            u32x4 cw_;
            auto load = [&](Frags& F, int slot) {
                const int so = slot * SLOT_BYTES;
#pragma unroll
                for (int j = 0; j < 4; ++j) F.b[j] = *(const LAS f16x8*)(bbase + so + j * 1024);
#pragma unroll
                for (int i = 0; i < 4; ++i) F.a[i] = *(const LAS f16x8*)(abase + so + i * 1024);
                if (HE) F.e = *(const LAS f16x8*)(ebase + so);
                cw_ = *(const volatile LAS u32x4*)ctlp;
            };
            auto mma = [&](const Frags& F) {
#pragma unroll
                for (int i = 0; i < 4; ++i)
#pragma unroll
                    for (int j = 0; j < 4; ++j)
                        acc[i][j] = __builtin_amdgcn_mfma_f32_16x16x32_f16(F.b[j], F.a[i], acc[i][j], 0, 0, 0);
                if (HE) {
#pragma unroll
                    for (int j = 0; j < 4; ++j) ex[j] = __builtin_amdgcn_mfma_f32_16x16x32_f16(F.b[j], F.e, ex[j], 0, 0, 0);
                }
            };
            auto step = [&](Frags& Fc, Frags& Fn, int s, int nslot) {
                __builtin_amdgcn_s_waitcnt(0xC07F);
                lds_st(done_addr, (unsigned)(s + 1));
                const int av = (int)__builtin_amdgcn_readfirstlane(min(min(cw_.x, cw_.y), cw_.z));
                if (av <= s + 1 && s + 1 < NSTEP) spin_avail(s + 1);
                __builtin_amdgcn_sched_barrier(0);
                load(Fn, nslot);
                mma(Fc);
#pragma unroll
                for (int g = 0; g < (HE ? 10 : 9); ++g) {
                    __builtin_amdgcn_sched_group_barrier(0x008, 1, 0);
                    __builtin_amdgcn_sched_group_barrier(0x100, 1, 0);
                }
                __builtin_amdgcn_sched_barrier(0);
            };
            spin_avail(0);
            load(F0, 0);
            int slot = 0;
            for (int s = 0; s < NSTEP; s += 2) {
                step(F0, F1, s, slot + 1);
                slot = (slot == NSLOT - 2) ? 0 : slot + 2;
                step(F1, F0, s + 1, slot);
            }
        };
        if (has_e) run(std::integral_constant<bool, true>{}); else run(std::integral_constant<bool, false>{});
        if (__builtin_amdgcn_readfirstlane(lds_ld4(ctl).w) != 0u) {
            const float nanv = __builtin_nanf("");
#pragma unroll
            for (int j = 0; j < 4; ++j) {
#pragma unroll
                for (int i = 0; i < 4; ++i) acc[i][j] = (f32x4){nanv, nanv, nanv, nanv};
                ex[j] = (f32x4){nanv, nanv, nanv, nanv};
            }
        }
        float* oc = p.out + jn * BN + wn_ * 64 + 4 * fq;
        float* op = oc + (size_t)(src * MS + half * BM + wm_ * 64 + fr16) * NS;
        float* oe = oc + (size_t)(rank * MS + BM + ((ef0 + ek) & 15) * 16 + fr16) * NS;
        auto blockout = [&](const f32x4 (&y)[4], float* dst) {
            f32x4 g[4];
#pragma unroll
            for (int j = 0; j < 4; ++j) g[j] = sigm4(y[j]);
#pragma unroll
            for (int r = 0; r < 4; ++r) {
                f32x4 v;
#pragma unroll
                for (int j = 0; j < 4; ++j) { float o = y[j][r] * g[j][r]; asm volatile("" : "+v"(o)); v[j] = o; }
                st_out(dst + 16 * r, v);
            }
        };
#pragma unroll
        for (int i = 0; i < 4; ++i) blockout(acc[i], op + (size_t)(16 * i) * NS);
        if (has_e) blockout(ex, oe);
        return;
    }

    __builtin_amdgcn_s_setprio(2);
    if (wave == NCONS) {
        const unsigned e1 = __builtin_amdgcn_readfirstlane(__hip_atomic_load(epw, __ATOMIC_RELAXED, __HIP_MEMORY_SCOPE_AGENT)) + 1u;
        if (lane < 32) lds_st(ctl + lane * 4, lane == 4 ? e1 : 0u);
        if (lane == 0) __hip_atomic_store(epw, e1, __ATOMIC_RELAXED, __HIP_MEMORY_SCOPE_AGENT);
        asm volatile("s_waitcnt lgkmcnt(0)" ::: "memory");
    }
    asm volatile("" ::: "memory");
    __builtin_amdgcn_s_barrier();
    asm volatile("" ::: "memory");
    const unsigned ep = __builtin_amdgcn_readfirstlane(lds_ld4(ctl + 16).x);
    unsigned* tmo = (unsigned*)(myws + TMO_OFF);
    if (wave == NCONS + 3) {
        if (im == 0) { watcher(p, myws, rank, lane, ep, jn); return; }
        const char* ldst = uni_ptr(myws + XS_OFF + ((size_t)(ep & 1u) * NR + rank) * XS_SLOT);
        unsigned* lflag = (unsigned*)(myws + FLAGX_OFF) + rank * 2 * NSTEP;
        int o = (im - 1) * 16 + jn;
        for (int st = NMT * CV_WAVES / 2 + im - 1; st < NSTEP; st += NMT - 1) {
            while (o < 2 * NSTEP && ((o >> 1) < st || st + NMT - 1 >= NSTEP)) {
                const int unit = (o & 1) * NSTEP + (o >> 1);
                const float* xu = p.x + (size_t)((o & 1) * BM + fr16) * KD + (o >> 1) * BK + 4 * fq;
#pragma unroll 1
                for (int hh = 0; hh < 2; ++hh) {
                    f32x4 xv[8][2];
#pragma unroll
                    for (int m = 0; m < 8; ++m) {
                        xv[m][0] = *(const f32x4*)(xu + (size_t)((hh * 8 + m) * 16) * KD);
                        xv[m][1] = *(const f32x4*)(xu + (size_t)((hh * 8 + m) * 16) * KD + 16);
                    }
                    f16x8 h[8];
#pragma unroll
                    for (int m = 0; m < 8; ++m)
#pragma unroll
                        for (int e = 0; e < 4; ++e) { h[m][e] = (_Float16)xv[m][0][e]; h[m][4 + e] = (_Float16)xv[m][1][e]; }
                    asm volatile("" : "+v"(h[0]), "+v"(h[1]), "+v"(h[2]), "+v"(h[3]), "+v"(h[4]), "+v"(h[5]), "+v"(h[6]), "+v"(h[7]) : : "memory");
#pragma unroll
                    for (int m = 0; m < 8; ++m) st16_dev(ldst, (unsigned)((unit * 16 + hh * 8 + m) * 1024 + lane * 16), h[m]);
                }
                wait_vm<0>();
                if (lane == 0) __hip_atomic_store(lflag + unit, ep, __ATOMIC_RELAXED, __HIP_MEMORY_SCOPE_AGENT);
                o += 14 * 16;
            }
            f32x4 wv[2][8];
#pragma unroll
            for (int pp = 0; pp < 2; ++pp)
#pragma unroll
                for (int e = 0; e < 8; ++e)
                    wv[pp][e] = ld_w(wl + (size_t)(BK * st + (e >> 2) * 16 + (e & 3)) * NT + 64 * pp);
            f16x8 h[8];
#pragma unroll
            for (int pp = 0; pp < 2; ++pp)
#pragma unroll
                for (int i = 0; i < 4; ++i)
#pragma unroll
                    for (int e = 0; e < 8; ++e) h[4 * pp + i][e] = (_Float16)wv[pp][e][i];
            asm volatile("" : "+v"(h[0]), "+v"(h[1]), "+v"(h[2]), "+v"(h[3]), "+v"(h[4]), "+v"(h[5]), "+v"(h[6]), "+v"(h[7]) : : "memory");
#pragma unroll
            for (int i = 0; i < 8; ++i) st16_dev(wdst, (unsigned)(st * B_BYTES + i * 1024 + lane * 16), h[i]);
            wait_vm<0>();
            if (lane < 2) __hip_atomic_store(fwp + lane * NSTEP + st, ep, __ATOMIC_RELAXED, __HIP_MEMORY_SCOPE_AGENT);
            __builtin_amdgcn_s_sleep(72);
        }
    } else if (wave == NCONS) {
        loader<false, false>(wdst, wdst, B_BYTES, lds_base, A_BYTES + E_BYTES, 0, ctl + 8, fwp, fwp + NSTEP, ep, tmo, lane, 0);
    } else {
        const int la = wave - NCONS - 1;
        const char* xsb = myws + XS_OFF + (size_t)(ep & 1u) * NR * XS_SLOT;
        const char* xs = xsb + (size_t)src * XS_SLOT + (size_t)half * NSTEP * A_BYTES + la * 8192;
        const char* xe = xsb + (size_t)rank * XS_SLOT + (size_t)NSTEP * A_BYTES + (ef0 + la) * 1024;
        const unsigned* fx = (const unsigned*)(myws + FLAGX_OFF);
        const unsigned* f0 = fx + (src * 2 + half) * NSTEP;
        if (la == 0 || im == 0)
            loader<true, true>(xs, xe, A_BYTES, lds_base, la * 8192, A_BYTES + la * 1024, ctl + 4 * la, f0, fx + (rank * 2 + 1) * NSTEP, ep, tmo, lane, 1 + la);
        else
            loader<true, false>(xs, xs, A_BYTES, lds_base, la * 8192, 0, ctl + 4 * la, f0, f0, ep, tmo, lane, 2);
    }
}

}

extern "C" void kernel_launch(int rank, int nranks, ncclComm_t comm,
                              hipStream_t stream,
                              void* const* d_in, const int* in_sizes, int n_in,
                              void* d_out, int out_size,
                              const int* in_dtypes, int out_dtype,
                              void* const* d_out_all,
                              void* const* d_ws_all, size_t ws_bytes) {
    (void)comm; (void)in_dtypes; (void)out_dtype; (void)d_out_all;
    if (nranks != NR || (unsigned)rank >= (unsigned)NR || n_in < 2 || ws_bytes < WS_NEED ||
        in_sizes[0] != MS * KD || in_sizes[1] != KD * NT || out_size != MT * NS) {
        fprintf(stderr, "kernel_launch: unexpected problem shape\n");
        return;
    }
    Params p;
    p.x = (const float*)d_in[0];
    p.w = (const float*)d_in[1];
    p.out = (float*)d_out;
    for (int i = 0; i < NR; ++i) p.ws[i] = (char*)d_ws_all[i];
    p.rank = rank;
    hipLaunchKernelGGL(fused_kernel, dim3(NBLK), dim3(NTHREADS), 0, stream, p);
    HIP_CHECK(hipGetLastError());
}
```

```cpp
#include <hip/hip_runtime.h>
#include <rccl/rccl.h>
#include <cstdio>
#include <cstdint>
#include <type_traits>

#define HIP_CHECK(expr) do { hipError_t _e = (expr); if (_e != hipSuccess) { \
    fprintf(stderr, "HIP error %s at %s:%d\n", hipGetErrorString(_e), __FILE__, __LINE__); \
    return; } } while (0)

namespace {

typedef float f32x4 __attribute__((ext_vector_type(4)));
typedef _Float16 f16x8 __attribute__((ext_vector_type(8)));
typedef unsigned u32x4 __attribute__((ext_vector_type(4)));
#define LAS __attribute__((address_space(3)))

constexpr int NR = 8;
constexpr int MT = 4096, KD = 4096, NT = 16384;
constexpr int MS = MT / NR;
constexpr int NS = NT / NR;
constexpr int BM = 256, BN = 128, BK = 32;
constexpr int NSTEP = KD / BK;
constexpr int NSLOT = 6;
constexpr int A_BYTES = BM * BK * 2;
constexpr int E_BYTES = 2048;
constexpr int B_BYTES = BN * BK * 2;
constexpr int SLOT_BYTES = A_BYTES + E_BYTES + B_BYTES;
constexpr int CTL_OFF = NSLOT * SLOT_BYTES;
constexpr int LDS_BYTES = CTL_OFF + 128;
constexpr int NCONS = 8;
constexpr int NWAVES = NCONS + 4;
constexpr int NTHREADS = NWAVES * 64;
constexpr int NBLK = 256;
constexpr int NCB = 16;
constexpr int NMT = 15;
constexpr int CWPB = 4;
constexpr int CWPB_KEEP = 3;
constexpr int RETIRE_F = 3584;
constexpr int NCW = NCB * CWPB;
constexpr int NFRAG = (MS / 16) * NSTEP;
constexpr int LDEPTH = 3;
constexpr int CV_WAVES = 2;

constexpr size_t EPOCH_OFF = 0;
constexpr size_t TMO_OFF = 16384;
constexpr size_t TICK_OFF = 16384 + 256;
constexpr size_t CNT_OFF = 32768;
constexpr size_t FLAGW_OFF = 65536;
constexpr size_t FLAGX_OFF = 65536 + 16384;
constexpr size_t XS_OFF = 1u << 20;
constexpr size_t XS_SLOT = (size_t)MS * KD * 2;
constexpr size_t W16_OFF = XS_OFF + 2 * NR * XS_SLOT;
constexpr size_t WS_NEED = W16_OFF + (size_t)KD * NS * 2;
constexpr unsigned long long DEADLINE_TICKS = 100000000ull;

struct Params {
    const float* x;
    const float* w;
    float* out;
    char* my;
    char* peer[2 * (NR - 1)];
    int rank;
};

__device__ __forceinline__ unsigned long long rt() { return __builtin_amdgcn_s_memrealtime(); }
__device__ __forceinline__ void lds_st(unsigned addr, unsigned v) {
    asm volatile("ds_write_b32 %0, %1" : : "v"(addr), "v"(v) : "memory");
}
__device__ __forceinline__ u32x4 lds_ld4(unsigned addr) {
    u32x4 v;
    asm volatile("ds_read_b128 %0, %1\n\ts_waitcnt lgkmcnt(0)" : "=&v"(v) : "v"(addr) : "memory");
    return v;
}
__device__ __forceinline__ void glds4k(const void* gsrc, unsigned lds_dst) {
    unsigned keep;
    asm volatile("s_mov_b32 %0, m0\n\ts_mov_b32 m0, %2\n\ts_nop 0\n\t"
                 "global_load_lds_dwordx4 %1, off sc1\n\t"
                 "global_load_lds_dwordx4 %1, off offset:1024 sc1\n\t"
                 "global_load_lds_dwordx4 %1, off offset:2048 sc1\n\t"
                 "global_load_lds_dwordx4 %1, off offset:3072 sc1\n\t"
                 "s_mov_b32 m0, %0"
                 : "=&s"(keep) : "v"(gsrc), "s"(lds_dst) : "memory");
}
__device__ __forceinline__ void glds1k(const void* gsrc, unsigned lds_dst) {
    unsigned keep;
    asm volatile("s_mov_b32 %0, m0\n\ts_mov_b32 m0, %2\n\ts_nop 0\n\t"
                 "global_load_lds_dwordx4 %1, off sc1\n\t"
                 "s_mov_b32 m0, %0"
                 : "=&s"(keep) : "v"(gsrc), "s"(lds_dst) : "memory");
}
template <int N> __device__ __forceinline__ void wait_vm() {
    asm volatile("s_waitcnt vmcnt(%0)" : : "i"(N) : "memory");
}
__device__ __forceinline__ void st16_sys(const char* base, unsigned off, f16x8 v) {
    asm volatile("global_store_dwordx4 %0, %1, %2 sc0 sc1\n\ts_nop 1" : : "v"(off), "v"(v), "s"(base) : "memory");
}
__device__ __forceinline__ void st16_dev(const char* base, unsigned off, f16x8 v) {
    asm volatile("global_store_dwordx4 %0, %1, %2 sc1\n\ts_nop 1" : : "v"(off), "v"(v), "s"(base) : "memory");
}
__device__ __forceinline__ void st_out(float* p, f32x4 v) {
    asm volatile("global_store_dwordx4 %0, %1, off sc1\n\ts_nop 1" : : "v"(p), "v"(v) : "memory");
}
__device__ __forceinline__ const char* uni_ptr(const char* p) {
    const unsigned long long u = (unsigned long long)p;
    const unsigned lo = __builtin_amdgcn_readfirstlane((unsigned)u), hi = __builtin_amdgcn_readfirstlane((unsigned)(u >> 32));
    return (const char*)(((unsigned long long)hi << 32) | lo);
}
__device__ __forceinline__ f32x4 ld_w(const float* p) { return __builtin_nontemporal_load((const f32x4*)p); }
__device__ __forceinline__ f32x4 sigm4(f32x4 y) {
    f32x4 c, e, r;
#pragma unroll
    for (int i = 0; i < 4; ++i) c[i] = __builtin_amdgcn_fmed3f(y[i], -60.0f, 60.0f);
    c = c * -1.44269504088896340736f;
#pragma unroll
    for (int i = 0; i < 4; ++i) e[i] = __builtin_amdgcn_exp2f(c[i]);
    e = e + 1.0f;
#pragma unroll
    for (int i = 0; i < 4; ++i) r[i] = __builtin_amdgcn_rcpf(e[i]);
    return r;
}

template <bool ISX, bool EXTRA>
__device__ __forceinline__ void loader(const char* src, const char* esrc, unsigned stride, unsigned lds_base,
                                       unsigned lds_off, unsigned elds_off, unsigned full_addr,
                                       const unsigned* flag0, const unsigned* flag1,
                                       unsigned ep, unsigned* tmo, int lane, int role) {
    constexpr int PIECES = 8 + (EXTRA ? 1 : 0);
    static_assert(PIECES * LDEPTH <= 60 && LDEPTH == 3, "vmcnt is 6 bits; retire ladder below assumes <= 2 unpublished");
    const unsigned ctl = lds_base + CTL_OFF;
    const char* lsrc = src + lane * 16;
    const char* le = esrc + lane * 16;
    const unsigned* pollp = (lane < 32 ? flag0 : flag1) + (lane & 31);
    int pub = 0, wm = 0, fr = NSLOT;
    int slot = 0;
    for (int s = 0; s < NSTEP; ++s) {
        if (s >= wm && role == 2) {
            while (true) {
                unsigned w;
                asm volatile("ds_read_b32 %0, %1 offset:24" : "=&v"(w) : "v"(ctl) : "memory");
                const bool had = pub < s;
                if (had) { wait_vm<0>(); lds_st(full_addr, (unsigned)s); pub = s; }
                asm volatile("s_waitcnt lgkmcnt(0)" : "+v"(w) : : "memory");
                wm = (int)__builtin_amdgcn_readfirstlane(w);
                if (wm > s) break;
                if (!had) __builtin_amdgcn_s_sleep(1);
            }
        }
        if (s >= wm) {
            const unsigned long long t0 = rt();
            const unsigned* pp = pollp + (s & ~31);
            unsigned spins = 0;
            while (true) {
                unsigned fl;
                if (ISX) asm volatile("global_load_dword %0, %1, off sc0 sc1" : "=&v"(fl) : "v"(pp) : "memory");
                else     asm volatile("global_load_dword %0, %1, off sc1" : "=&v"(fl) : "v"(pp) : "memory");
                if (pub < s) { wait_vm<1>(); lds_st(full_addr, (unsigned)s); pub = s; }
                asm volatile("s_waitcnt vmcnt(0)" : "+v"(fl) : : "memory");
                const unsigned long long m = __ballot((int)(fl - ep) >= 0);
                const unsigned both = (unsigned)m & (unsigned)(m >> 32);
                wm = (s & ~31) + (both == 0xffffffffu ? 32 : __builtin_ctz(~both));
                if (wm > s) break;
                if ((++spins & 63u) == 0u) {
                    const unsigned tv = __builtin_amdgcn_readfirstlane(__hip_atomic_load(tmo, __ATOMIC_RELAXED, __HIP_MEMORY_SCOPE_AGENT));
                    if (tv == ep || rt() - t0 > DEADLINE_TICKS) {
                        if (lane == 0) __hip_atomic_store(tmo, ep, __ATOMIC_RELAXED, __HIP_MEMORY_SCOPE_AGENT);
                        lds_st(ctl + 12, 1u);
                        wait_vm<0>();
                        wm = NSTEP;
                        break;
                    }
                }
                if (s < NSTEP - 12) __builtin_amdgcn_s_sleep(32); else __builtin_amdgcn_s_sleep(2);
            }
            if (role == 1) lds_st(ctl + 24, (unsigned)wm);
        }
        if (s >= fr) {
            while (true) {
                const u32x4 d0 = lds_ld4(ctl + 64), d1 = lds_ld4(ctl + 80);
                unsigned mn = min(min(min(d0.x, d0.y), min(d0.z, d0.w)), min(min(d1.x, d1.y), min(d1.z, d1.w)));
                fr = (int)__builtin_amdgcn_readfirstlane(mn) + NSLOT;
                if (fr > s) break;
                if (pub < s) {
                    if (s - pub == 1) wait_vm<0>(); else wait_vm<PIECES>();
                    ++pub; lds_st(full_addr, (unsigned)pub);
                } else {
                    __builtin_amdgcn_s_sleep(1);
                }
            }
        }
        const unsigned dst = lds_base + (unsigned)slot * SLOT_BYTES;
        const size_t go = (size_t)s * stride;
        glds4k(lsrc + go, dst + lds_off);
        glds4k(lsrc + go + 4096, dst + lds_off + 4096);
        if (EXTRA) glds1k(le + go, dst + elds_off);
        slot = (slot == NSLOT - 1) ? 0 : slot + 1;
        if (s + 1 - pub >= LDEPTH) { wait_vm<PIECES * (LDEPTH - 1)>(); ++pub; lds_st(full_addr, (unsigned)pub); }
    }
    wait_vm<0>();
    lds_st(full_addr, (unsigned)NSTEP);
}

__device__ __forceinline__ void watcher(const Params& p, char* myws, int rank, int lane, unsigned ep, int w) {
    unsigned* tmo = (unsigned*)(myws + TMO_OFF);
    const unsigned* cntp = (const unsigned*)(myws + CNT_OFF);
    unsigned* flagp = (unsigned*)(p.peer[lane & (NR - 1)] + FLAGX_OFF) + rank * 2 * NSTEP;
    const unsigned need = ep * 16u;
    const unsigned long long t0 = rt();
    unsigned spins = 0;
    for (unsigned o = w; o < 2u * NSTEP; o += 16) {
        while (true) {
            unsigned c;
            asm volatile("global_load_dword %0, %1, off sc1\n\ts_waitcnt vmcnt(0)" : "=&v"(c) : "v"(cntp + o * 32u) : "memory");
            if ((int)(__builtin_amdgcn_readfirstlane(c) - need) >= 0) break;
            if ((++spins & 255u) == 0u && rt() - t0 > DEADLINE_TICKS) {
                if (lane == 0) __hip_atomic_store(tmo, ep, __ATOMIC_RELAXED, __HIP_MEMORY_SCOPE_AGENT);
                return;
            }
            __builtin_amdgcn_s_sleep(1);
        }
        if (lane < NR - 1)
            asm volatile("global_store_dword %0, %1, off sc0 sc1" : : "v"(flagp + (o & 1u) * NSTEP + (o >> 1)), "v"(ep) : "memory");
    }
}

struct Frags { f16x8 a[4], b[4], e; };

__global__ void __launch_bounds__(NTHREADS, 3) fused_kernel(Params p) {
    __shared__ __attribute__((aligned(1024))) char lds[LDS_BYTES];
    const int tid = threadIdx.x;
    const int lane = tid & 63;
    const int wave = __builtin_amdgcn_readfirstlane(tid >> 6);
    const int blk = blockIdx.x;
    const int rank = p.rank & (NR - 1);
    char* const myws = p.my;
    const int fr16 = lane & 15, fq = lane >> 4;
    unsigned* const epw = (unsigned*)(myws + EPOCH_OFF) + blk * 16;
    const unsigned lds_base = (unsigned)(size_t)(LAS char*)lds;

    if (blk < NCB) {
        if (wave >= CWPB) return;
        const int cw = wave * NCB + blk;
        const float* xl = p.x + (size_t)fr16 * KD + 4 * fq;
        auto ld = [&](f32x4& a, f32x4& b, unsigned f) {
            f = min(f, (unsigned)NFRAG - 1u);
            const float* xp = xl + (size_t)((f & 31u) * 16u) * KD + (f >> 5) * BK;
            asm volatile("global_load_dwordx4 %0, %2, off\n\tglobal_load_dwordx4 %1, %2, off offset:64" : "=&v"(a), "=&v"(b) : "v"(xp) : "memory");
        };
        unsigned epv;
        asm volatile("global_load_dword %0, %1, off sc1" : "=&v"(epv) : "v"(epw + wave) : "memory");
        f32x4 a0, a1, b0, b1;
        ld(a0, a1, (unsigned)cw);
        ld(b0, b1, (unsigned)(cw + NCW));
        const char* dst[NR - 1];
        {
            const int rot = cw % (NR - 1);
#pragma unroll
            for (int q = 0; q < NR - 1; ++q) dst[q] = p.peer[rot + q];
            asm volatile("" : "+s"(dst[0]), "+s"(dst[1]), "+s"(dst[2]), "+s"(dst[3]), "+s"(dst[4]), "+s"(dst[5]), "+s"(dst[6]));
        }
        asm volatile("s_waitcnt vmcnt(2)" : "+v"(epv), "+v"(a0), "+v"(a1) : : "memory");
        const unsigned ep = __builtin_amdgcn_readfirstlane(epv) + 1u;
        if (lane == 0) __hip_atomic_store(epw + wave, ep, __ATOMIC_RELAXED, __HIP_MEMORY_SCOPE_AGENT);
        const size_t slot_off = XS_OFF + ((size_t)(ep & 1u) * NR + rank) * XS_SLOT;
#pragma unroll
        for (int q = 0; q < NR - 1; ++q) dst[q] += slot_off;
        unsigned* tick = (unsigned*)(myws + TICK_OFF) + (ep & 1u) * 32u;
        if (cw == 0 && lane == 0)
            __hip_atomic_store((unsigned*)(myws + TICK_OFF) + ((ep + 1u) & 1u) * 32u, 0u, __ATOMIC_RELAXED, __HIP_MEMORY_SCOPE_AGENT);
        unsigned* cntp = (unsigned*)(myws + CNT_OFF);
        const unsigned flim = wave < CWPB_KEEP ? (unsigned)NFRAG : (unsigned)RETIRE_F;
        unsigned f0 = (unsigned)cw, f1 = f0 + NCW, f2 = f1 + NCW, fp = 0xffffffffu;
        auto count = [&](unsigned f) {
            if (lane == 0) asm volatile("global_atomic_add %0, %1, off" : : "v"(cntp + (f >> 4) * 32u), "v"(1u) : "memory");
        };
        auto round = [&](f32x4& c0, f32x4& c1, f32x4& n0, f32x4& n1) {
            f16x8 h;
#pragma unroll
            for (int e = 0; e < 4; ++e) { h[e] = (_Float16)c0[e]; h[4 + e] = (_Float16)c1[e]; }
            asm volatile("" : "+v"(h) : : "memory");
            unsigned t = 0;
            const bool draw = f2 < flim;
            {
                unsigned long long save;
                const unsigned msk = __builtin_amdgcn_readfirstlane(draw ? 1u : 0u);
                asm volatile("s_mov_b64 %1, exec\n\ts_mov_b32 exec_lo, %4\n\ts_mov_b32 exec_hi, 0\n\tglobal_atomic_add %0, %2, %3, off sc0\n\ts_mov_b64 exec, %1"
                             : "+v"(t), "=&s"(save) : "v"(tick), "v"(1u), "s"(msk) : "memory");
            }
            const unsigned off = ((((f0 >> 4) & 1u) * NSTEP + (f0 >> 5)) * 16u + (f0 & 15u)) * 1024u + (unsigned)lane * 16u;
#pragma unroll
            for (int q = 0; q < NR - 1; ++q) st16_sys(dst[q], off, h);
            ld(c0, c1, f2);
            asm volatile("s_waitcnt vmcnt(9)" : "+v"(t), "+v"(n0), "+v"(n1) : : "memory");
            if (fp != 0xffffffffu) count(fp);
            fp = f0; f0 = f1; f1 = f2;
            f2 = draw ? __builtin_amdgcn_readfirstlane(t) + 3u * NCW : 0xffffffffu;
        };
        while (true) {
            round(a0, a1, b0, b1);
            if (f0 >= (unsigned)NFRAG) break;
            round(b0, b1, a0, a1);
            if (f0 >= (unsigned)NFRAG) break;
        }
        asm volatile("s_waitcnt vmcnt(0)" : "+v"(a0), "+v"(a1), "+v"(b0), "+v"(b1) : : "memory");
        count(fp);
        return;
    }

    const unsigned ctl = lds_base + CTL_OFF;
    const int xcd = blk & 7;
    const int t = (xcd >> 1) * 30 + ((blk >> 3) - 2);
    const int im = t >> 3;
    const int jn = (xcd & 1) * 8 + (t & 7);
    const int src = (rank + ((im + 1) >> 1)) & (NR - 1);
    const int half = (im == 0) ? 0 : ((im + 1) & 1);
    const int ef0 = (im == 0) ? 14 : im - 1;
    const float* wl = p.w + (size_t)(4 * fq) * NT + (size_t)rank * NS + jn * BN + 4 * (4 * (fr16 & 3) + (fr16 >> 2));
    const char* wdst = uni_ptr(myws + W16_OFF + (size_t)jn * NSTEP * B_BYTES);
    unsigned* fwp = (unsigned*)(myws + FLAGW_OFF) + jn * 2 * NSTEP;

    if (wave < NCONS) {
        const int cst = im + NMT * (wave >> 1), cp = wave & 1;
        f32x4 wv[8];
        if (wave < CV_WAVES) {
            const float* wu = wl + (size_t)(BK * cst) * NT + 64 * cp;
#pragma unroll
            for (int e = 0; e < 8; ++e) wv[e] = ld_w(wu + (size_t)((e >> 2) * 16 + (e & 3)) * NT);
        }
        asm volatile("" ::: "memory");
        __builtin_amdgcn_s_barrier();
        asm volatile("" ::: "memory");
        const unsigned ep = __builtin_amdgcn_readfirstlane(lds_ld4(ctl + 16).x);
        if (wave < CV_WAVES) {
            f16x8 h[4];
#pragma unroll
            for (int i = 0; i < 4; ++i)
#pragma unroll
                for (int e = 0; e < 8; ++e) h[i][e] = (_Float16)wv[e][i];
            asm volatile("" : "+v"(h[0]), "+v"(h[1]), "+v"(h[2]), "+v"(h[3]) : : "memory");
#pragma unroll
            for (int i = 0; i < 4; ++i) st16_dev(wdst, (unsigned)(cst * B_BYTES + cp * 4096 + i * 1024 + lane * 16), h[i]);
            wait_vm<0>();
            if (lane == 0) __hip_atomic_store(fwp + cp * NSTEP + cst, ep, __ATOMIC_RELAXED, __HIP_MEMORY_SCOPE_AGENT);
        }
        const int wm_ = wave & 3, wn_ = wave >> 2;
        const int ek = wm_ >> 1;
        const bool has_e = ((wm_ & 1) == wn_) && (ek == 0 || im == 0);
        f32x4 acc[4][4], ex[4];
#pragma unroll
        for (int j = 0; j < 4; ++j) {
#pragma unroll
            for (int i = 0; i < 4; ++i) acc[i][j] = (f32x4){0.f, 0.f, 0.f, 0.f};
            ex[j] = (f32x4){0.f, 0.f, 0.f, 0.f};
        }
        const LAS char* abase = (const LAS char*)lds + wm_ * 4096 + lane * 16;
        const LAS char* ebase = (const LAS char*)lds + A_BYTES + ek * 1024 + lane * 16;
        const LAS char* bbase = (const LAS char*)lds + A_BYTES + E_BYTES + wn_ * 4096 + lane * 16;
        const LAS u32x4* ctlp = (const LAS u32x4*)((const LAS char*)lds + CTL_OFF);
        const unsigned done_addr = ctl + 64 + wave * 4;
        auto spin_avail = [&](int s) {
            while (true) {
                const u32x4 f = lds_ld4(ctl);
                if ((int)__builtin_amdgcn_readfirstlane(min(min(f.x, f.y), f.z)) > s) break;
                __builtin_amdgcn_s_sleep(1);
            }
        };
        auto run = [&](auto he) {
            constexpr bool HE = decltype(he)::value;
            Frags F0, F1;
            u32x4 cw_;
            auto load = [&](Frags& F, int slot) {
                const int so = slot * SLOT_BYTES;
#pragma unroll
                for (int j = 0; j < 4; ++j) F.b[j] = *(const LAS f16x8*)(bbase + so + j * 1024);
#pragma unroll
                for (int i = 0; i < 4; ++i) F.a[i] = *(const LAS f16x8*)(abase + so + i * 1024);
                if (HE) F.e = *(const LAS f16x8*)(ebase + so);
                cw_ = *(const volatile LAS u32x4*)ctlp;
            };
            auto mma = [&](const Frags& F) {
#pragma unroll
                for (int i = 0; i < 4; ++i)
#pragma unroll
                    for (int j = 0; j < 4; ++j)
                        acc[i][j] = __builtin_amdgcn_mfma_f32_16x16x32_f16(F.b[j], F.a[i], acc[i][j], 0, 0, 0);
                if (HE) {
#pragma unroll
                    for (int j = 0; j < 4; ++j) ex[j] = __builtin_amdgcn_mfma_f32_16x16x32_f16(F.b[j], F.e, ex[j], 0, 0, 0);
                }
            };
            auto step = [&](Frags& Fc, Frags& Fn, int s, int nslot) {
                __builtin_amdgcn_s_waitcnt(0xC07F);
                lds_st(done_addr, (unsigned)(s + 1));
                const int av = (int)__builtin_amdgcn_readfirstlane(min(min(cw_.x, cw_.y), cw_.z));
                if (av <= s + 1 && s + 1 < NSTEP) spin_avail(s + 1);
                __builtin_amdgcn_sched_barrier(0);
                load(Fn, nslot);
                mma(Fc);
#pragma unroll
                for (int g = 0; g < (HE ? 10 : 9); ++g) {
                    __builtin_amdgcn_sched_group_barrier(0x008, 1, 0);
                    __builtin_amdgcn_sched_group_barrier(0x100, 1, 0);
                }
                __builtin_amdgcn_sched_barrier(0);
            };
            spin_avail(0);
            load(F0, 0);
            int slot = 0;
            for (int s = 0; s < NSTEP; s += 2) {
                step(F0, F1, s, slot + 1);
                slot = (slot == NSLOT - 2) ? 0 : slot + 2;
                step(F1, F0, s + 1, slot);
            }
        };
        if (has_e) run(std::integral_constant<bool, true>{}); else run(std::integral_constant<bool, false>{});
        if (__builtin_amdgcn_readfirstlane(lds_ld4(ctl).w) != 0u) {
            const float nanv = __builtin_nanf("");
#pragma unroll
            for (int j = 0; j < 4; ++j) {
#pragma unroll
                for (int i = 0; i < 4; ++i) acc[i][j] = (f32x4){nanv, nanv, nanv, nanv};
                ex[j] = (f32x4){nanv, nanv, nanv, nanv};
            }
        }
        float* oc = p.out + jn * BN + wn_ * 64 + 4 * fq;
        float* op = oc + (size_t)(src * MS + half * BM + wm_ * 64 + fr16) * NS;
        float* oe = oc + (size_t)(rank * MS + BM + ((ef0 + ek) & 15) * 16 + fr16) * NS;
        auto blockout = [&](const f32x4 (&y)[4], float* dst) {
            f32x4 g[4];
#pragma unroll
            for (int j = 0; j < 4; ++j) g[j] = sigm4(y[j]);
#pragma unroll
            for (int r = 0; r < 4; ++r) {
                f32x4 v;
#pragma unroll
                for (int j = 0; j < 4; ++j) { float o = y[j][r] * g[j][r]; asm volatile("" : "+v"(o)); v[j] = o; }
                st_out(dst + 16 * r, v);
            }
        };
#pragma unroll
        for (int i = 0; i < 4; ++i) blockout(acc[i], op + (size_t)(16 * i) * NS);
        if (has_e) blockout(ex, oe);
        return;
    }

    __builtin_amdgcn_s_setprio(2);
    if (wave == NCONS) {
        const unsigned e1 = __builtin_amdgcn_readfirstlane(__hip_atomic_load(epw, __ATOMIC_RELAXED, __HIP_MEMORY_SCOPE_AGENT)) + 1u;
        if (lane < 32) lds_st(ctl + lane * 4, lane == 4 ? e1 : 0u);
        if (lane == 0) __hip_atomic_store(epw, e1, __ATOMIC_RELAXED, __HIP_MEMORY_SCOPE_AGENT);
        asm volatile("s_waitcnt lgkmcnt(0)" ::: "memory");
    }
    asm volatile("" ::: "memory");
    __builtin_amdgcn_s_barrier();
    asm volatile("" ::: "memory");
    const unsigned ep = __builtin_amdgcn_readfirstlane(lds_ld4(ctl + 16).x);
    unsigned* tmo = (unsigned*)(myws + TMO_OFF);
    if (wave == NCONS + 3) {
        if (im == 0) { watcher(p, myws, rank, lane, ep, jn); return; }
        const char* ldst = uni_ptr(myws + XS_OFF + ((size_t)(ep & 1u) * NR + rank) * XS_SLOT);
        unsigned* lflag = (unsigned*)(myws + FLAGX_OFF) + rank * 2 * NSTEP;
        int o = (im - 1) * 16 + jn;
        const unsigned long long th0 = rt();
        auto pace = [&](int due) {
            const unsigned long long tgt = th0 + (unsigned long long)(max(due, 0) * 40);
            while (rt() < tgt) __builtin_amdgcn_s_sleep(8);
        };
        for (int st = NMT * CV_WAVES / 2 + im - 1; st < NSTEP; st += NMT - 1) {
            while (o < 2 * NSTEP && ((o >> 1) < st || st + NMT - 1 >= NSTEP)) {
                pace(min((o >> 1) - 10, st - 15));
                const int unit = (o & 1) * NSTEP + (o >> 1);
                const float* xu = p.x + (size_t)((o & 1) * BM + fr16) * KD + (o >> 1) * BK + 4 * fq;
#pragma unroll 1
                for (int hh = 0; hh < 2; ++hh) {
                    f32x4 xv[8][2];
#pragma unroll
                    for (int m = 0; m < 8; ++m) {
                        xv[m][0] = *(const f32x4*)(xu + (size_t)((hh * 8 + m) * 16) * KD);
                        xv[m][1] = *(const f32x4*)(xu + (size_t)((hh * 8 + m) * 16) * KD + 16);
                    }
                    f16x8 h[8];
#pragma unroll
                    for (int m = 0; m < 8; ++m)
#pragma unroll
                        for (int e = 0; e < 4; ++e) { h[m][e] = (_Float16)xv[m][0][e]; h[m][4 + e] = (_Float16)xv[m][1][e]; }
                    asm volatile("" : "+v"(h[0]), "+v"(h[1]), "+v"(h[2]), "+v"(h[3]), "+v"(h[4]), "+v"(h[5]), "+v"(h[6]), "+v"(h[7]) : : "memory");
#pragma unroll
                    for (int m = 0; m < 8; ++m) st16_dev(ldst, (unsigned)((unit * 16 + hh * 8 + m) * 1024 + lane * 16), h[m]);
                }
                wait_vm<0>();
                if (lane == 0) __hip_atomic_store(lflag + unit, ep, __ATOMIC_RELAXED, __HIP_MEMORY_SCOPE_AGENT);
                o += 14 * 16;
            }
            pace(st - 15);
            f32x4 wv[2][8];
#pragma unroll
            for (int pp = 0; pp < 2; ++pp)
#pragma unroll
                for (int e = 0; e < 8; ++e)
                    wv[pp][e] = ld_w(wl + (size_t)(BK * st + (e >> 2) * 16 + (e & 3)) * NT + 64 * pp);
            f16x8 h[8];
#pragma unroll
            for (int pp = 0; pp < 2; ++pp)
#pragma unroll
                for (int i = 0; i < 4; ++i)
#pragma unroll
                    for (int e = 0; e < 8; ++e) h[4 * pp + i][e] = (_Float16)wv[pp][e][i];
            asm volatile("" : "+v"(h[0]), "+v"(h[1]), "+v"(h[2]), "+v"(h[3]), "+v"(h[4]), "+v"(h[5]), "+v"(h[6]), "+v"(h[7]) : : "memory");
#pragma unroll
            for (int i = 0; i < 8; ++i) st16_dev(wdst, (unsigned)(st * B_BYTES + i * 1024 + lane * 16), h[i]);
            wait_vm<0>();
            if (lane < 2) __hip_atomic_store(fwp + lane * NSTEP + st, ep, __ATOMIC_RELAXED, __HIP_MEMORY_SCOPE_AGENT);
        }
    } else if (wave == NCONS) {
        loader<false, false>(wdst, wdst, B_BYTES, lds_base, A_BYTES + E_BYTES, 0, ctl + 8, fwp, fwp + NSTEP, ep, tmo, lane, 0);
    } else {
        const int la = wave - NCONS - 1;
        const char* xsb = myws + XS_OFF + (size_t)(ep & 1u) * NR * XS_SLOT;
        const char* xs = xsb + (size_t)src * XS_SLOT + (size_t)half * NSTEP * A_BYTES + la * 8192;
        const char* xe = xsb + (size_t)rank * XS_SLOT + (size_t)NSTEP * A_BYTES + (ef0 + la) * 1024;
        const unsigned* fx = (const unsigned*)(myws + FLAGX_OFF);
        const unsigned* f0 = fx + (src * 2 + half) * NSTEP;
        if (la == 0 || im == 0)
            loader<true, true>(xs, xe, A_BYTES, lds_base, la * 8192, A_BYTES + la * 1024, ctl + 4 * la, f0, fx + (rank * 2 + 1) * NSTEP, ep, tmo, lane, 1 + la);
        else
            loader<true, false>(xs, xs, A_BYTES, lds_base, la * 8192, 0, ctl + 4 * la, f0, f0, ep, tmo, lane, 2);
    }
}

}

extern "C" void kernel_launch(int rank, int nranks, ncclComm_t comm,
                              hipStream_t stream,
                              void* const* d_in, const int* in_sizes, int n_in,
                              void* d_out, int out_size,
                              const int* in_dtypes, int out_dtype,
                              void* const* d_out_all,
                              void* const* d_ws_all, size_t ws_bytes) {
    (void)comm; (void)in_dtypes; (void)out_dtype; (void)d_out_all;
    if (nranks != NR || (unsigned)rank >= (unsigned)NR || n_in < 2 || ws_bytes < WS_NEED ||
        in_sizes[0] != MS * KD || in_sizes[1] != KD * NT || out_size != MT * NS) {
        fprintf(stderr, "kernel_launch: unexpected problem shape\n");
        return;
    }
    Params p;
    p.x = (const float*)d_in[0];
    p.w = (const float*)d_in[1];
    p.out = (float*)d_out;
    p.my = (char*)d_ws_all[rank];
    for (int q = 0; q < 2 * (NR - 1); ++q) p.peer[q] = (char*)d_ws_all[(rank + 1 + q % (NR - 1)) % NR];
    p.rank = rank;
    hipLaunchKernelGGL(fused_kernel, dim3(NBLK), dim3(NTHREADS), 0, stream, p);
    HIP_CHECK(hipGetLastError());
}
```
